```python
import numpy as np

kernel_name = "gpu_ag_gemm_m2048_k2048_n2048_bf16_relu"


ABSMAX_RTOL = 0.008

COMPUTE_DTYPE = "bfloat16"
EPILOGUE = "relu"
IN_QUANT = None
OUT_QUANT = None
IN_DTYPES = {"x": "bfloat16", "w_mat": "bfloat16"}
WORLD_SIZE = 8

def _to_bf16(x):
    u = np.ascontiguousarray(x, dtype=np.float32).view(np.uint32)
    return ((u + 0x7FFF + ((u >> 16) & 1)) & 0xFFFF0000).view(np.float32)

def setup_inputs():

    x = _to_bf16(np.random.randn(2048, 2048).astype(np.float32))
    w_mat = _to_bf16((np.random.randn(2048, 2048) / np.sqrt(2048)).astype(np.float32))
    return {"x": x, "w_mat": w_mat}

def _epilogue(y):
    return np.maximum(y, 0.0).astype(np.float32)

def reference(x, w_mat):
    y = (x @ w_mat).astype(np.float32)
    return _epilogue(y)
```

```cpp
#include <hip/hip_runtime.h>
#include <rccl/rccl.h>
#include <cstdio>

#define HIP_CHECK(expr) do { hipError_t _e = (expr); if (_e != hipSuccess) { \
    fprintf(stderr, "HIP error %s at %s:%d\n", hipGetErrorString(_e), __FILE__, __LINE__); \
    return; } } while (0)

#define MODE_AG 0
#define ACQ_MODE 1
#define LD_AUX 0

namespace {

constexpr int NR = 8;
constexpr int KDIM = 2048;
constexpr int MS = 256, NS = 256;
constexpr int BM = 32, BN = 64, KC = 128, NC = KDIM / KC;
constexpr int NWG = 256;
constexpr int NPWG = 32;
constexpr int NWAVES = 8;
constexpr int NTHREADS = NWAVES * 64;

constexpr int NMT = 32;
constexpr int PPC = 70;
constexpr int NPAIRS = PPC * NC;
constexpr int NP = NPWG * NWAVES;
constexpr int MAXP = (NPAIRS + NP - 1) / NP;

constexpr size_t XT_BYTES = (size_t)BM * KDIM * 2;
constexpr size_t WT_BYTES = (size_t)BN * KDIM * 2;
constexpr size_t XSTG_BYTES = (size_t)NMT * XT_BYTES;
constexpr size_t WSTG_BYTES = MODE_AG ? 0 : 4 * WT_BYTES;
constexpr size_t PAR_BYTES = XSTG_BYTES + WSTG_BYTES;
constexpr size_t FLAG_OFF = 16u << 20;
static_assert(2 * PAR_BYTES <= FLAG_OFF, "staging overflows");
constexpr size_t WS_NEED = FLAG_OFF + (256u << 10);
constexpr int FX_W = 0;
constexpr int FW_W = 2048;
constexpr int FY_W = 4096;
constexpr int CNT_W = 8192;
constexpr int TMO_W = 8192 + 1024;
constexpr int DBG_W = 12288;

constexpr unsigned long long DEADLINE = 50000000ull;

constexpr int XS_BYTES = BM * KC * 2;
constexpr int WS_BYTES = BN * KC * 2;
constexpr int NSLOT = 6;
constexpr int LDS_X = 0;
constexpr int LDS_W = NSLOT * XS_BYTES;
constexpr int LDS_CTL = LDS_W + NSLOT * WS_BYTES;
constexpr int LDS_BYTES = LDS_CTL + 64;

typedef __attribute__((ext_vector_type(4))) unsigned u32x4;
typedef __attribute__((ext_vector_type(2))) unsigned u32x2;
typedef __attribute__((ext_vector_type(8))) short bf16x8;
typedef __attribute__((ext_vector_type(4))) float f32x4;
typedef __attribute__((ext_vector_type(2))) float f32x2;
typedef __attribute__((ext_vector_type(2))) __bf16 bf16x2;
typedef __attribute__((address_space(3))) unsigned lds_u32;

struct Params {
  const char* x;
  const char* w;
  char* out[NR];
  char* ws[NR];
  int rank;
};

__device__ __forceinline__ void st16_sys(void* p, u32x4 v) {
  asm volatile("global_store_dwordx4 %0, %1, off sc0 sc1\n\ts_nop 1" :: "v"(p), "v"(v) : "memory");
}
__device__ __forceinline__ void st4_sys(void* p, unsigned v) {
  asm volatile("global_store_dword %0, %1, off sc0 sc1" :: "v"(p), "v"(v) : "memory");
}
__device__ __forceinline__ unsigned ld4_sys(const void* p) {
  return __hip_atomic_load((const unsigned*)p, __ATOMIC_RELAXED, __HIP_MEMORY_SCOPE_SYSTEM);
}
__device__ __forceinline__ unsigned lds_ld(unsigned* p) {
  return __hip_atomic_load(p, __ATOMIC_RELAXED, __HIP_MEMORY_SCOPE_WORKGROUP);
}
__device__ __forceinline__ void lds_st(unsigned* p, unsigned v) {
  __hip_atomic_store(p, v, __ATOMIC_RELAXED, __HIP_MEMORY_SCOPE_WORKGROUP);
}
#define WAITV(n) asm volatile("s_waitcnt vmcnt(" #n ")" ::: "memory")

template <int AUX>
__device__ __forceinline__ void dma16(const char* g, unsigned l) {
  if constexpr (AUX == 0)
    asm volatile("s_mov_b32 m0, %1\n\ts_nop 0\n\tglobal_load_lds_dwordx4 %0, off" :: "v"(g), "s"(l) : "memory");
  else if constexpr (AUX == 16)
    asm volatile("s_mov_b32 m0, %1\n\ts_nop 0\n\tglobal_load_lds_dwordx4 %0, off sc1" :: "v"(g), "s"(l) : "memory");
  else if constexpr (AUX == 17)
    asm volatile("s_mov_b32 m0, %1\n\ts_nop 0\n\tglobal_load_lds_dwordx4 %0, off sc0 sc1" :: "v"(g), "s"(l) : "memory");
  else
    asm volatile("s_mov_b32 m0, %1\n\ts_nop 0\n\tglobal_load_lds_dwordx4 %0, off nt" :: "v"(g), "s"(l) : "memory");
}

__device__ __forceinline__ unsigned pack_relu(float a, float b) {
  a = a < 0.f ? 0.f : a;
  b = b < 0.f ? 0.f : b;
  f32x2 f = {a, b};
  bf16x2 h = __builtin_convertvector(f, bf16x2);
  return __builtin_bit_cast(unsigned, h);
}

#define STAMP(idx) do {} while (0)

struct Tile { int mt, xsrc, xt, owner, ct, wslot; };

__device__ __forceinline__ bool get_tile(int rank, int j, int which, Tile& T) {
  const int xcd = j & 7, s = j >> 3;
  if (which == 1 && s >= 4) return false;
  const int nt = xcd;
  const int par_me = rank & 1;
  T.wslot = nt - 4;
  T.owner = nt < 4 ? rank : (1 - par_me) + 2 * (nt - 4);
  T.ct = nt < 4 ? nt : (rank >> 1);
  const int lbase = (rank >> 1) * 5;
  int mt;
  if (which == 0 && s < 5) mt = lbase + s;
  else {
    const int r = (which == 1) ? 23 + s : s - 5;
    mt = r < lbase ? r : r + 5;
  }
  T.mt = mt;
  if (mt < 20) { T.xsrc = par_me + 2 * (mt / 5); T.xt = mt % 5; }
  else { T.xsrc = (1 - par_me) + 2 * ((mt - 20) / 3); T.xt = 5 + (mt - 20) % 3; }
  return true;
}

__global__ void __launch_bounds__(NTHREADS, 2) ag_gemm_kernel(Params P) {
  __shared__ __attribute__((aligned(1024))) char smem[LDS_BYTES];
  unsigned* ctl = (unsigned*)(smem + LDS_CTL);
  const int tid = threadIdx.x, lane = tid & 63;
  const int wv = __builtin_amdgcn_readfirstlane(tid >> 6);
  const int b = blockIdx.x;
  const int rank = P.rank;
  char* myws = P.ws[rank];
  unsigned* myf = (unsigned*)(myws + FLAG_OFF);
  unsigned* tmo = myf + TMO_W;

  if (wv == 0) STAMP(0);

  if (b < NPWG) {
    unsigned call0 = 0;
    if (tid == 0) { call0 = myf[CNT_W + b] + 1; myf[CNT_W + b] = call0; lds_st(ctl + 0, call0); }
    const int g = b * NWAVES + wv;
    u32x4 v[MAXP][4];
    unsigned doff[MAXP], foff[MAXP];
    int dst[MAXP];
    const int r = lane >> 3;
#pragma unroll
    for (int j = 0; j < MAXP; ++j) {
      const int i = g + j * NP;
      if (i < NPAIRS) {
        const int c = i / PPC, q = i % PPC;
        const int d = 1 + q % 7, u = q / 7;
        const int dest = (rank + d) & 7;
        dst[j] = dest;
        bool isw = false; int xu = u;
        if (d & 1) { if (u < 4) isw = true; else xu = 10 + (u - 4); }
        if (!isw) {
          const int t = xu >> 1, h = xu & 1;
          const int dmt = t < 5 ? (rank >> 1) * 5 + t : 20 + (rank >> 1) * 3 + (t - 5);
#pragma unroll
          for (int rr = 0; rr < 2; ++rr) {
            const int row16 = rr * 8 + r;
            const int c16 = (lane & 7) ^ ((row16 >> 1) & 7);
            const char* s = P.x + (size_t)(t * 32 + h * 16 + row16) * (KDIM * 2) + c * (KC * 2) + c16 * 16;
            v[j][rr * 2 + 0] = *(const u32x4*)(s);
            v[j][rr * 2 + 1] = *(const u32x4*)(s + 128);
          }
          doff[j] = (unsigned)(dmt * XT_BYTES) + c * XS_BYTES + h * 4096;
          foff[j] = FX_W + (dmt * NC + c) * 2 + h;
        } else {
          const int dct = dest >> 1;
#pragma unroll
          for (int kk = 0; kk < 4; ++kk) {
            const int kb = u * 4 + kk;
            const int fw = ((r >> 1) & 1) | ((kb & 1) << 2);
            const int c16 = (lane & 7) ^ fw;
            v[j][kk] = *(const u32x4*)(P.w + (size_t)(c * KC + kb * 8 + r) * (NS * 2) + dct * 128 + c16 * 16);
          }
          doff[j] = (unsigned)(XSTG_BYTES + (rank >> 1) * WT_BYTES) + c * WS_BYTES + u * 4096;
          foff[j] = FW_W + ((rank >> 1) * NC + c) * 4 + u;
        }
      }
    }
    asm volatile("s_waitcnt lgkmcnt(0)" ::: "memory");
    __builtin_amdgcn_s_barrier();
    asm volatile("" ::: "memory");
    const unsigned call = lds_ld(ctl + 0);
    const size_t poff = (size_t)(call & 1) * PAR_BYTES;
    WAITV(0);
    if (wv == 0) STAMP(6);
#pragma unroll
    for (int j = 0; j < MAXP; ++j) {
      const int i = g + j * NP;
      if (i < NPAIRS) {
        char* dp = P.ws[dst[j]] + poff + doff[j] + lane * 16;
#pragma unroll
        for (int ks = 0; ks < 4; ++ks) st16_sys(dp + ks * 1024, v[j][ks]);
        WAITV(0);
        if (lane == 0) st4_sys((unsigned*)(P.ws[dst[j]] + FLAG_OFF) + foff[j], call);
      }
    }
    if (wv == 0) STAMP(1);
    if (b == 0) {
      const unsigned long long t0 = __builtin_amdgcn_s_memrealtime();
      bool done = false, bad = false;
      while (!done) {
        const unsigned fy = ld4_sys(myf + FY_W + wv * 64 + lane);
        const unsigned tm = __hip_atomic_load(tmo, __ATOMIC_RELAXED, __HIP_MEMORY_SCOPE_AGENT);
        done = __all(fy >= call);
        if (!done) {
          if (tm == call || __builtin_amdgcn_s_memrealtime() - t0 > DEADLINE) {
            if (lane == 0) __hip_atomic_store(tmo, call, __ATOMIC_RELAXED, __HIP_MEMORY_SCOPE_AGENT);
            bad = true; done = true;
          } else __builtin_amdgcn_s_sleep(1);
        }
      }
      if (bad && lane == 0) {
        ((unsigned short*)P.out[rank])[(size_t)(rank * MS) * NS + wv] = 0x7fc0;
      }
      if (wv == 0) STAMP(3);
    }
    return;
  }

  const int j = b - NPWG;
  if (wv > 4) return;

  if (wv == 4) {
    unsigned call = 0;
    if (lane == 0) {
      call = myf[CNT_W + b] + 1;
      myf[CNT_W + b] = call;
      lds_st(ctl + 0, call);
      lds_st(ctl + 1, 0u);
      lds_st(ctl + 3, 0u);
      lds_st(ctl + 4, 0u);
    }
    call = __builtin_amdgcn_readfirstlane(call);
    asm volatile("s_waitcnt lgkmcnt(0)" ::: "memory");
    __builtin_amdgcn_s_barrier();
    const unsigned long long t0 = __builtin_amdgcn_s_memrealtime();
    bool bad = false;
    for (int which = 0; which < 2; ++which) {
      Tile T;
      if (!get_tile(rank, j, which, T)) break;
      const bool xrem = T.xsrc != rank, wrem = T.owner != rank;
      int rdy = 0;
      if ((!xrem && !wrem) || bad) { rdy = NC; if (lane == 0) lds_st(ctl + 1, (unsigned)(which * NC + NC)); }
      while (rdy < NC) {
        unsigned fx = call, fw = call;
        if (xrem && lane < 32) fx = ld4_sys(myf + FX_W + T.mt * 32 + lane);
        if (wrem) fw = ld4_sys(myf + FW_W + T.wslot * 64 + lane);
        const unsigned tm = __hip_atomic_load(tmo, __ATOMIC_RELAXED, __HIP_MEMORY_SCOPE_AGENT);
        unsigned long long bx = __ballot(fx >= call), bw = __ballot(fw >= call);
        bx &= bx >> 1; bx &= 0x5555555555555555ull;
        bw &= bw >> 1; bw &= bw >> 2; bw &= 0x1111111111111111ull;
        const int nrx = __builtin_ctzll((~bx & 0x5555555555555555ull) | (1ull << 32)) >> 1;
        const unsigned long long nb = ~bw & 0x1111111111111111ull;
        const int nrw = nb ? (__builtin_ctzll(nb) >> 2) : NC;
        int nr = nrx < nrw ? nrx : nrw;
        if (tm == call || __builtin_amdgcn_s_memrealtime() - t0 > DEADLINE) {
          if (lane == 0) { __hip_atomic_store(tmo, call, __ATOMIC_RELAXED, __HIP_MEMORY_SCOPE_AGENT); lds_st(ctl + 4, 1u); }
          bad = true; nr = NC;
        }
        if (nr > rdy) {
          __builtin_amdgcn_fence(__ATOMIC_ACQUIRE, "");
          asm volatile("s_waitcnt vmcnt(0)" ::: "memory");
          if (lane == 0) lds_st(ctl + 1, (unsigned)(which * NC + nr));
          rdy = nr;
        } else {
          __builtin_amdgcn_s_sleep(1);
        }
      }
    }
    STAMP(2);
    return;
  }

  const int wm = wv >> 1, wn = wv & 1;
  const unsigned lbase = (unsigned)(size_t)(__attribute__((address_space(3))) char*)smem;
  const int r8 = lane >> 3;
  __builtin_amdgcn_s_barrier();
  asm volatile("" ::: "memory");
  const unsigned call = lds_ld(ctl + 0);
  const char* sbase = myws + (size_t)(call & 1) * PAR_BYTES;

  const int m16 = lane & 15, g4 = lane >> 4, q4 = (lane & 15) >> 2, p4 = lane & 3;
  const unsigned xa0 = (2 * wm + (m16 >> 3)) * 2048 + (m16 & 7) * 128 + ((g4 ^ ((m16 >> 1) & 7)) * 16);
  const unsigned xa1 = xa0 ^ 64;
  const unsigned fwl = ((q4 >> 1) & 1) | ((g4 & 1) << 2);
  const unsigned wa_0 = g4 * 1024 + q4 * 128 + (((wn * 4 + p4) ^ fwl) * 16) + ((0 ^ (p4 & 1)) * 8);
  const unsigned wa_1 = wa_0 ^ 8;

  for (int which = 0; which < 2; ++which) {
    Tile T;
    if (!get_tile(rank, j, which, T)) break;
    const bool xrem = T.xsrc != rank, wrem = T.owner != rank;
    const char* xg; unsigned xstride_c, xstride_ks;
    if (xrem) {
      xg = sbase + (size_t)T.mt * XT_BYTES + wv * 2048 + lane * 16; xstride_c = XS_BYTES; xstride_ks = 1024;
    } else {
      const int row16 = (wv & 1) * 8 + r8;
      const int c16 = (lane & 7) ^ ((row16 >> 1) & 7);
      xg = P.x + (size_t)(T.xt * 32 + wv * 8 + r8) * (KDIM * 2) + c16 * 16; xstride_c = KC * 2; xstride_ks = 128;
    }
    const char* wg0; const char* wg1; unsigned wstride_c, wstride_k2;
    if (wrem) {
      wg0 = sbase + XSTG_BYTES + (size_t)T.wslot * WT_BYTES + wv * 4096 + lane * 16; wg1 = wg0 + 1024;
      wstride_c = WS_BYTES; wstride_k2 = 2048;
    } else {
      const int f0 = ((r8 >> 1) & 1), f1 = f0 | 4;
      wg0 = P.w + (size_t)(wv * 32 + r8) * (NS * 2) + T.ct * 128 + (((lane & 7) ^ f0) * 16);
      wg1 = P.w + (size_t)(wv * 32 + 8 + r8) * (NS * 2) + T.ct * 128 + (((lane & 7) ^ f1) * 16);
      wstride_c = KC * NS * 2; wstride_k2 = 16 * NS * 2;
    }
    f32x4 acc0 = {0.f, 0.f, 0.f, 0.f}, acc1 = {0.f, 0.f, 0.f, 0.f};
    const int cbase = which * NC;
    int issued = 0;
    for (int c = 0; c < NC; ++c) {
      int rdy = (int)lds_ld(ctl + 1) - cbase;
      while (issued <= c && rdy <= issued) { __builtin_amdgcn_s_sleep(0); rdy = (int)lds_ld(ctl + 1) - cbase; }
      asm volatile("" ::: "memory");
      int lim = c + NSLOT - 1;
      lim = lim < rdy ? lim : rdy;
      lim = lim < NC ? lim : NC;
      while (issued < lim) {
        const int sl = (cbase + issued) % NSLOT;
        const unsigned lw = lbase + LDS_W + sl * WS_BYTES + wv * 4096;
        const unsigned lx = lbase + LDS_X + sl * XS_BYTES + wv * 2048;
        const char* w0 = wg0 + (size_t)issued * wstride_c;
        const char* w1 = wg1 + (size_t)issued * wstride_c;
        const char* x0 = xg + (size_t)issued * xstride_c;
        dma16<LD_AUX>(w0, lw);
        dma16<LD_AUX>(w1, lw + 1024);
        dma16<LD_AUX>(w0 + wstride_k2, lw + 2048);
        dma16<LD_AUX>(w1 + wstride_k2, lw + 3072);
        dma16<LD_AUX>(x0, lx);
        dma16<LD_AUX>(x0 + xstride_ks, lx + 1024);
        ++issued;
      }
      switch (issued - c - 1) {
        case 0: WAITV(0); break;
        case 1: WAITV(6); break;
        case 2: WAITV(12); break;
        case 3: WAITV(18); break;
        default: WAITV(24); break;
      }
      asm volatile("" ::: "memory");
      if (lane == 0) __hip_atomic_fetch_add(ctl + 3, 1u, __ATOMIC_RELAXED, __HIP_MEMORY_SCOPE_WORKGROUP);
      while (lds_ld(ctl + 3) < 4u * (cbase + c + 1)) { __builtin_amdgcn_s_sleep(0); }
      asm volatile("" ::: "memory");
      {
        const int sl = (cbase + c) % NSLOT;
        const unsigned xb = lbase + LDS_X + sl * XS_BYTES;
        const unsigned wb = lbase + LDS_W + sl * WS_BYTES;
        const unsigned ax0 = xb + xa0, ax1 = xb + xa1, aw0 = wb + wa_0, aw1 = wb + wa_1;
        u32x4 xf[2][2]; u32x2 wf[2][2][2][2];
#define LOADG(B, G) asm volatile( \
      "ds_read_b128 %0, %10 offset:" #G "*1024\n\t" \
      "ds_read_b64_tr_b16 %2, %12 offset:" #G "*8192\n\t" \
      "ds_read_b64_tr_b16 %3, %12 offset:" #G "*8192+512\n\t" \
      "ds_read_b64_tr_b16 %4, %13 offset:" #G "*8192\n\t" \
      "ds_read_b64_tr_b16 %5, %13 offset:" #G "*8192+512\n\t" \
      "ds_read_b128 %1, %11 offset:" #G "*1024\n\t" \
      "ds_read_b64_tr_b16 %6, %12 offset:" #G "*8192+4096\n\t" \
      "ds_read_b64_tr_b16 %7, %12 offset:" #G "*8192+4608\n\t" \
      "ds_read_b64_tr_b16 %8, %13 offset:" #G "*8192+4096\n\t" \
      "ds_read_b64_tr_b16 %9, %13 offset:" #G "*8192+4608\n\t" \
      : "=&v"(xf[B][0]), "=&v"(xf[B][1]), "=&v"(wf[B][0][0][0]), "=&v"(wf[B][0][0][1]), "=&v"(wf[B][0][1][0]), "=&v"(wf[B][0][1][1]), \
        "=&v"(wf[B][1][0][0]), "=&v"(wf[B][1][0][1]), "=&v"(wf[B][1][1][0]), "=&v"(wf[B][1][1][1]) \
      : "v"(ax0), "v"(ax1), "v"(aw0), "v"(aw1) : "memory")
#define WAITG(B, N) asm volatile("s_waitcnt lgkmcnt(" #N ")" \
      : "+v"(xf[B][0]), "+v"(xf[B][1]), "+v"(wf[B][0][0][0]), "+v"(wf[B][0][0][1]), "+v"(wf[B][0][1][0]), "+v"(wf[B][0][1][1]), \
        "+v"(wf[B][1][0][0]), "+v"(wf[B][1][0][1]), "+v"(wf[B][1][1][0]), "+v"(wf[B][1][1][1]) :: "memory")
#define MMAG(B) do { _Pragma("unroll") for (int s = 0; s < 2; ++s) { \
      u32x4 a0 = {wf[B][s][0][0][0], wf[B][s][0][0][1], wf[B][s][0][1][0], wf[B][s][0][1][1]}; \
      u32x4 a1 = {wf[B][s][1][0][0], wf[B][s][1][0][1], wf[B][s][1][1][0], wf[B][s][1][1][1]}; \
      acc0 = __builtin_amdgcn_mfma_f32_16x16x32_bf16(__builtin_bit_cast(bf16x8, a0), __builtin_bit_cast(bf16x8, xf[B][s]), acc0, 0, 0, 0); \
      acc1 = __builtin_amdgcn_mfma_f32_16x16x32_bf16(__builtin_bit_cast(bf16x8, a1), __builtin_bit_cast(bf16x8, xf[B][s]), acc1, 0, 0, 0); } } while (0)
        LOADG(0, 0);
        LOADG(1, 1);
        WAITG(0, 10); MMAG(0);
        WAITG(1, 0); MMAG(1);
      }
    }
    const bool bad = lds_ld(ctl + 4) != 0;
    const f32x4 lo = (g4 & 1) ? acc1 : acc0, hi = (g4 & 1) ? acc0 : acc1;
    u32x4 o;
    o[0] = pack_relu(lo[0], lo[1]); o[1] = pack_relu(lo[2], lo[3]);
    o[2] = pack_relu(hi[0], hi[1]); o[3] = pack_relu(hi[2], hi[3]);
    if (bad) { o[0] = o[1] = o[2] = o[3] = 0x7fc07fc0u; }
    const int grow = T.xsrc * MS + T.xt * 32 + wm * 16 + m16;
    char* op = P.out[T.owner] + (size_t)grow * (NS * 2) + T.ct * 128 + wn * 64 + g4 * 16;
    if (!wrem) {
      *(u32x4*)op = o;
    } else {
      st16_sys(op, o);
      WAITV(0);
      if (lane == 0) st4_sys((unsigned*)(P.ws[T.owner] + FLAG_OFF) + FY_W + ((rank >> 1) * 32 + T.mt) * 4 + wv, call);
    }
    if (wv == 0) STAMP(4 + which);
  }
}

}

extern "C" void kernel_launch(int rank, int nranks, ncclComm_t comm,
                              hipStream_t stream,
                              void* const* d_in, const int* in_sizes, int n_in,
                              void* d_out, int out_size,
                              const int* in_dtypes, int out_dtype,
                              void* const* d_out_all,
                              void* const* d_ws_all, size_t ws_bytes) {
  if (nranks != NR || ws_bytes < WS_NEED || n_in < 2 || (unsigned)rank >= (unsigned)NR) {
    fprintf(stderr, "unsupported configuration\n");
    return;
  }
  Params p;
  p.x = (const char*)d_in[0];
  p.w = (const char*)d_in[1];
  for (int i = 0; i < NR; ++i) { p.out[i] = (char*)d_out_all[i]; p.ws[i] = (char*)d_ws_all[i]; }
  p.rank = rank;
  ag_gemm_kernel<<<NWG, NTHREADS, 0, stream>>>(p);
  HIP_CHECK(hipGetLastError());
}
```

```cpp
#include <hip/hip_runtime.h>
#include <rccl/rccl.h>
#include <cstdio>
#include <type_traits>

#define HIP_CHECK(expr) do { hipError_t _e = (expr); if (_e != hipSuccess) { \
    fprintf(stderr, "HIP error %s at %s:%d\n", hipGetErrorString(_e), __FILE__, __LINE__); \
    return; } } while (0)

#define MODE_AG 0
#define ACQ_MODE 2
#define PFD 5
#define PUSH_AFTER_FENCE 1
#define NPW 1
#define NPUSHWG 256
#define XMAP 1
#define PIPE 1
#define LD_AUX 0

namespace {

constexpr int NR = 8;
constexpr int KDIM = 2048;
constexpr int MS = 256, NS = 256;
constexpr int BM = 32, BN = 64, KC = 128, NC = KDIM / KC;
constexpr int NWG = 256;
constexpr int NWAVES = 5 + NPW;
constexpr int NTHREADS = NWAVES * 64;

constexpr int NMT = 32;
constexpr int PPC = 70;
constexpr int NPAIRS = PPC * NC;
constexpr int NP = NPUSHWG * NPW;
static_assert(NPAIRS >= 2 * NP, "two static units per pusher wave");

constexpr size_t XT_BYTES = (size_t)BM * KDIM * 2;
constexpr size_t WT_BYTES = (size_t)BN * KDIM * 2;
constexpr size_t XSTG_BYTES = (size_t)NMT * XT_BYTES;
constexpr size_t WSTG_BYTES = MODE_AG ? 0 : 4 * WT_BYTES;
constexpr size_t PAR_BYTES = XSTG_BYTES + WSTG_BYTES;
constexpr size_t FLAG_OFF = 16u << 20;
static_assert(2 * PAR_BYTES <= FLAG_OFF, "staging overflows");
constexpr size_t WS_NEED = FLAG_OFF + (256u << 10);
constexpr int FX_W = 0;
constexpr int FW_W = 2048;
constexpr int FY_W = 4096;
constexpr int CNT_W = 8192;
constexpr int TMO_W = 8192 + 1024;
constexpr int TICKET_W = 8192 + 2048;
constexpr int DBG_W = 12288;

constexpr unsigned long long DEADLINE = 50000000ull;

constexpr int XS_BYTES = BM * KC * 2;
constexpr int WS_BYTES = BN * KC * 2;
constexpr int NSLOT = 6;
constexpr int LDS_X = 0;
constexpr int LDS_W = NSLOT * XS_BYTES;
constexpr int LDS_CTL = LDS_W + NSLOT * WS_BYTES;
constexpr int LDS_BYTES = LDS_CTL + 64;

typedef __attribute__((ext_vector_type(4))) unsigned u32x4;
typedef __attribute__((ext_vector_type(2))) unsigned u32x2;
typedef __attribute__((ext_vector_type(8))) short bf16x8;
typedef __attribute__((ext_vector_type(4))) float f32x4;
typedef __attribute__((ext_vector_type(2))) float f32x2;
typedef __attribute__((ext_vector_type(2))) __bf16 bf16x2;
typedef __attribute__((address_space(3))) unsigned lds_u32;

struct Params {
  const char* x;
  const char* w;
  char* out[NR];
  char* ws[NR];
  int rank;
};

__device__ __forceinline__ void st16_sys(void* p, u32x4 v) {
  asm volatile("global_store_dwordx4 %0, %1, off sc0 sc1\n\ts_nop 1" :: "v"(p), "v"(v) : "memory");
}
__device__ __forceinline__ void st4_sys(void* p, unsigned v) {
  asm volatile("global_store_dword %0, %1, off sc0 sc1" :: "v"(p), "v"(v) : "memory");
}
__device__ __forceinline__ unsigned ld4_sys(const void* p) {
  return __hip_atomic_load((const unsigned*)p, __ATOMIC_RELAXED, __HIP_MEMORY_SCOPE_SYSTEM);
}
__device__ __forceinline__ unsigned lds_ld(unsigned* p) {
  return __hip_atomic_load(p, __ATOMIC_RELAXED, __HIP_MEMORY_SCOPE_WORKGROUP);
}
__device__ __forceinline__ void lds_st(unsigned* p, unsigned v) {
  __hip_atomic_store(p, v, __ATOMIC_RELAXED, __HIP_MEMORY_SCOPE_WORKGROUP);
}
#define WAITV(n) asm volatile("s_waitcnt vmcnt(" #n ")" ::: "memory")

template <int AUX>
__device__ __forceinline__ void dma16(const char* g, unsigned l) {
  if constexpr (AUX == 0)
    asm volatile("s_mov_b32 m0, %1\n\ts_nop 0\n\tglobal_load_lds_dwordx4 %0, off" :: "v"(g), "s"(l) : "memory");
  else if constexpr (AUX == 16)
    asm volatile("s_mov_b32 m0, %1\n\ts_nop 0\n\tglobal_load_lds_dwordx4 %0, off sc1" :: "v"(g), "s"(l) : "memory");
  else if constexpr (AUX == 17)
    asm volatile("s_mov_b32 m0, %1\n\ts_nop 0\n\tglobal_load_lds_dwordx4 %0, off sc0 sc1" :: "v"(g), "s"(l) : "memory");
  else
    asm volatile("s_mov_b32 m0, %1\n\ts_nop 0\n\tglobal_load_lds_dwordx4 %0, off nt" :: "v"(g), "s"(l) : "memory");
}

__device__ __forceinline__ unsigned pack_relu(float a, float b) {
  a = a < 0.f ? 0.f : a;
  b = b < 0.f ? 0.f : b;
  f32x2 f = {a, b};
  bf16x2 h = __builtin_convertvector(f, bf16x2);
  return __builtin_bit_cast(unsigned, h);
}

#define STAMP(idx) do {} while (0)

struct Tile { int mt, xsrc, xt, owner, ct, wslot; };

__device__ __forceinline__ void get_tile(int rank, int b, Tile& T) {
  const int xcd = b & 7, s = b >> 3;
  const int nt = (xcd >> 2) * 4 + (s >> 3);
  const int mt = (xcd & 3) * 8 + (s & 7);
  const int par_me = rank & 1;
  T.wslot = nt - 4;
  T.owner = nt < 4 ? rank : (1 - par_me) + 2 * (nt - 4);
  T.ct = nt < 4 ? nt : (rank >> 1);
  T.mt = mt;
  if (mt < 20) { T.xsrc = par_me + 2 * (mt / 5); T.xt = mt % 5; }
  else { T.xsrc = (1 - par_me) + 2 * ((mt - 20) / 3); T.xt = 5 + (mt - 20) % 3; }
}

__global__ void __launch_bounds__(NTHREADS, 1) ag_gemm_kernel(Params P) {
  __shared__ __attribute__((aligned(1024))) char smem[LDS_BYTES];
  unsigned* ctl = (unsigned*)(smem + LDS_CTL);
  const int tid = threadIdx.x, lane = tid & 63;
  const int wv = __builtin_amdgcn_readfirstlane(tid >> 6);
  const int b = blockIdx.x;
  const int rank = P.rank;
  char* myws = P.ws[rank];
  unsigned* myf = (unsigned*)(myws + FLAG_OFF);
  unsigned* tmo = myf + TMO_W;

  if (wv == 0) STAMP(0);

  if (wv >= 5) {
    const int g = b * NPW + (wv - 5);
    __builtin_amdgcn_s_barrier();
    if (b >= NPUSHWG) return;
    unsigned call = myf[CNT_W + NWG + g] + 1;
    const int r = lane >> 3;
    const int r16 = lane >> 2;
    const unsigned xl = r16 * (KDIM * 2) + ((lane & 3) ^ ((r16 >> 2) & 3)) * 16;
    const unsigned wl0 = r * (NS * 2) + ((lane & 7) ^ ((r >> 1) & 1)) * 16;
    const unsigned wl1 = (r + 8) * (NS * 2) + ((lane & 7) ^ (((r >> 1) & 1) | 4)) * 16;
    struct Unit { u32x4 v[4]; char* dp; unsigned* fp; };
    auto fetch = [&](int i, Unit& U) {
      const int c = i / PPC, q = i % PPC;
      const int d = 1 + q % 7, u = q / 7;
      const int dest = (rank + d) & 7;
      char* dws = P.ws[dest];
      bool isw = false; int xu = u;
      if (d & 1) { if (u < 4) isw = true; else xu = 10 + (u - 4); }
      const int t = xu >> 1, h = xu & 1;
      const int dmt = t < 5 ? (rank >> 1) * 5 + t : 20 + (rank >> 1) * 3 + (t - 5);
      const char* sb = isw ? P.w + (size_t)(c * KC + u * 32) * (NS * 2) + (dest >> 1) * 128
                           : P.x + (size_t)(t * 32) * (KDIM * 2) + c * (KC * 2) + h * 128;
      const unsigned o0 = isw ? wl0 : xl, o1 = isw ? wl1 : xl + 16 * KDIM * 2;
      const unsigned o2 = isw ? wl0 + 16 * NS * 2 : xl + 64, o3 = isw ? wl1 + 16 * NS * 2 : xl + 64 + 16 * KDIM * 2;
      U.v[0] = *(const u32x4*)(sb + o0);
      U.v[1] = *(const u32x4*)(sb + o1);
      U.v[2] = *(const u32x4*)(sb + o2);
      U.v[3] = *(const u32x4*)(sb + o3);
      U.dp = dws + (isw ? (unsigned)(XSTG_BYTES + (rank >> 1) * WT_BYTES) + c * WS_BYTES + u * 4096
                        : (unsigned)(dmt * XT_BYTES) + c * XS_BYTES + h * 4096) + lane * 16;
      U.fp = (unsigned*)(dws + FLAG_OFF) + (isw ? FW_W + ((rank >> 1) * NC + c) * 4 + u : FX_W + (dmt * NC + c) * 2 + h);
    };
    Unit UA, UB;
    fetch(g, UA);
    fetch(g + NP, UB);
    if (lane == 0) myf[CNT_W + NWG + g] = call;
    const size_t poff = (size_t)(call & 1) * PAR_BYTES;
    const unsigned tbase = (call - 1) * (unsigned)(NPAIRS - NP) - 2u * NP;
    STAMP(6);
    int nxt = g + NP;
    auto step = [&](Unit& U) -> bool {
      char* dp = U.dp + poff;
#pragma unroll
      for (int ks = 0; ks < 4; ++ks) st16_sys(dp + ks * 1024, U.v[ks]);
      unsigned tk = 0;
      const bool more = nxt < NPAIRS;
      if (more && lane == 0) tk = __hip_atomic_fetch_add(myf + TICKET_W, 1u, __ATOMIC_RELAXED, __HIP_MEMORY_SCOPE_AGENT);
      WAITV(0);
      if (lane == 0) st4_sys(U.fp, call);
      if (!more) return false;
      nxt = (int)(__builtin_amdgcn_readfirstlane(tk) - tbase);
      if (nxt < NPAIRS) fetch(nxt, U);
      return true;
    };
    while (true) {
      if (!step(UA)) break;
      if (!step(UB)) break;
    }
    STAMP(1);
    return;
  }

  if (wv == 4) {
    unsigned call = 0;
    if (lane < 8) lds_st(ctl + lane, 0u);
    asm volatile("s_waitcnt lgkmcnt(0)" ::: "memory");
    __builtin_amdgcn_s_barrier();
    if (lane == 0) {
      call = myf[CNT_W + b] + 1;
      myf[CNT_W + b] = call;
    }
    call = __builtin_amdgcn_readfirstlane(call);
    __builtin_amdgcn_fence(__ATOMIC_ACQUIRE, "");
    asm volatile("s_waitcnt vmcnt(0)" ::: "memory");
    if (lane == 0) lds_st(ctl + 0, call);
    const unsigned long long t0 = __builtin_amdgcn_s_memrealtime();
    bool bad = false;
    {
      Tile T;
      get_tile(rank, b, T);
      const bool xrem = T.xsrc != rank, wrem = T.owner != rank;
      int rdy = 0;
      if (!xrem && !wrem) { rdy = NC; if (lane == 0) lds_st(ctl + 1, (unsigned)NC); }
      unsigned it = 0;
      while (rdy < NC) {
        unsigned f = call;
        const int cx = rdy + (lane >> 1), cw = rdy + ((lane - 8) >> 2);
        if (lane < 8) { if (xrem && cx < NC) f = ld4_sys(myf + FX_W + T.mt * 32 + rdy * 2 + lane); }
        else if (lane < 24) { if (wrem && cw < NC) f = ld4_sys(myf + FW_W + T.wslot * 64 + rdy * 4 + (lane - 8)); }
        unsigned tm = 0;
        if ((++it & 15) == 0) tm = __hip_atomic_load(tmo, __ATOMIC_RELAXED, __HIP_MEMORY_SCOPE_AGENT);
        const unsigned bl = (unsigned)__ballot(f >= call);
        unsigned bx = bl & 0xffu, bw = (bl >> 8) & 0xffffu;
        bx &= bx >> 1; bx &= 0x55u;
        bw &= bw >> 1; bw &= bw >> 2; bw &= 0x1111u;
        const int nrx = __builtin_ctz((~bx & 0x55u) | 0x100u) >> 1;
        const int nrw = __builtin_ctz((~bw & 0x1111u) | 0x10000u) >> 2;
        int nr = rdy + (nrx < nrw ? nrx : nrw);
        nr = nr < NC ? nr : NC;
        if (tm == call || ((it & 15) == 0 && __builtin_amdgcn_s_memrealtime() - t0 > DEADLINE)) {
          if (lane == 0) { __hip_atomic_store(tmo, call, __ATOMIC_RELAXED, __HIP_MEMORY_SCOPE_AGENT); lds_st(ctl + 4, 1u); }
          bad = true; nr = NC;
        }
        if (nr > rdy) {
          if (lane == 0) lds_st(ctl + 1, (unsigned)nr);
          rdy = nr;
        } else {
          __builtin_amdgcn_s_sleep(1);
        }
      }
    }
    STAMP(2);
    if (b < 8) {
      bool done = false;
      while (!done) {
        const unsigned fy = ld4_sys(myf + FY_W + b * 64 + lane);
        const unsigned tm = __hip_atomic_load(tmo, __ATOMIC_RELAXED, __HIP_MEMORY_SCOPE_AGENT);
        done = __all(fy >= call);
        if (!done) {
          if (tm == call || __builtin_amdgcn_s_memrealtime() - t0 > DEADLINE) {
            if (lane == 0) __hip_atomic_store(tmo, call, __ATOMIC_RELAXED, __HIP_MEMORY_SCOPE_AGENT);
            bad = true; done = true;
          } else __builtin_amdgcn_s_sleep(1);
        }
      }
      if (bad && lane == 0) {
        ((unsigned short*)P.out[rank])[(size_t)(rank * MS) * NS + b] = 0x7fc0;
      }
      STAMP(3);
    }
    return;
  }

  const int wm = wv >> 1, wn = wv & 1;
  const unsigned lbase = (unsigned)(size_t)(__attribute__((address_space(3))) char*)smem;
  const int r8 = lane >> 3;
  __builtin_amdgcn_s_barrier();
  asm volatile("" ::: "memory");
  unsigned call = lds_ld(ctl + 0);
  while (call == 0) { __builtin_amdgcn_s_sleep(0); call = lds_ld(ctl + 0); }
  asm volatile("" ::: "memory");
  const char* sbase = myws + (size_t)(call & 1) * PAR_BYTES;

  const int m16 = lane & 15, g4 = lane >> 4, q4 = (lane & 15) >> 2, p4 = lane & 3;
  const unsigned xa0 = wv * 2048 + m16 * 64 + ((g4 ^ ((m16 >> 2) & 3)) * 16);
  const unsigned fwl = ((q4 >> 1) & 1) | ((g4 & 1) << 2);
  const unsigned wa_0 = wv * 4096 + g4 * 1024 + q4 * 128 + ((p4 ^ fwl) * 16) + ((p4 & 1) * 8);

  {
    Tile T;
    get_tile(rank, b, T);
    const bool xrem = T.xsrc != rank, wrem = T.owner != rank;
    const char* xg; unsigned xstride_c, xstride_ks;
    if (xrem) {
      xg = sbase + (size_t)T.mt * XT_BYTES + wv * 2048 + lane * 16; xstride_c = XS_BYTES; xstride_ks = 1024;
    } else {
      const int r16 = lane >> 2;
      const int c4 = (lane & 3) ^ ((r16 >> 2) & 3);
      xg = P.x + (size_t)(T.xt * 32 + r16) * (KDIM * 2) + wv * 64 + c4 * 16; xstride_c = KC * 2; xstride_ks = 16 * KDIM * 2;
    }
    const char* wg0; const char* wg1; unsigned wstride_c, wstride_k2;
    if (wrem) {
      wg0 = sbase + XSTG_BYTES + (size_t)T.wslot * WT_BYTES + wv * 4096 + lane * 16; wg1 = wg0 + 1024;
      wstride_c = WS_BYTES; wstride_k2 = 2048;
    } else {
      const int f0 = ((r8 >> 1) & 1), f1 = f0 | 4;
      wg0 = P.w + (size_t)(wv * 32 + r8) * (NS * 2) + T.ct * 128 + (((lane & 7) ^ f0) * 16);
      wg1 = P.w + (size_t)(wv * 32 + 8 + r8) * (NS * 2) + T.ct * 128 + (((lane & 7) ^ f1) * 16);
      wstride_c = KC * NS * 2; wstride_k2 = 16 * NS * 2;
    }
    f32x4 acc[2][2][2];
#pragma unroll
    for (int i = 0; i < 8; ++i) acc[i >> 2][(i >> 1) & 1][i & 1] = f32x4{0.f, 0.f, 0.f, 0.f};
    constexpr int cbase = 0;
    int issued = 0;
    for (int c = 0; c < NC; ++c) {
      int rdy = (int)lds_ld(ctl + 1) - cbase;
      while (issued <= c && rdy <= issued) { __builtin_amdgcn_s_sleep(0); rdy = (int)lds_ld(ctl + 1) - cbase; }
      asm volatile("" ::: "memory");
      int lim = c + PFD;
      lim = lim < rdy ? lim : rdy;
      lim = lim < NC ? lim : NC;
      while (issued < lim) {
        const int sl = (cbase + issued) % NSLOT;
        const unsigned lw = lbase + LDS_W + sl * WS_BYTES + wv * 4096;
        const unsigned lx = lbase + LDS_X + sl * XS_BYTES + wv * 2048;
        const char* w0 = wg0 + (size_t)issued * wstride_c;
        const char* w1 = wg1 + (size_t)issued * wstride_c;
        const char* x0 = xg + (size_t)issued * xstride_c;
        {
        dma16<LD_AUX>(w0, lw);
        dma16<LD_AUX>(w1, lw + 1024);
        dma16<LD_AUX>(w0 + wstride_k2, lw + 2048);
        dma16<LD_AUX>(w1 + wstride_k2, lw + 3072);
        dma16<LD_AUX>(x0, lx);
        dma16<LD_AUX>(x0 + xstride_ks, lx + 1024);
        }
        ++issued;
      }
      switch (issued - c - 1) {
        case 0: WAITV(0); break;
        case 1: WAITV(6); break;
        case 2: WAITV(12); break;
        case 3: WAITV(18); break;
        default: WAITV(24); break;
      }
      {
        const int sl = c % NSLOT;
        const unsigned ax = lbase + LDS_X + sl * XS_BYTES + xa0;
        const unsigned aw = lbase + LDS_W + sl * WS_BYTES + wa_0;
        const unsigned aw00 = aw, aw01 = aw ^ 8, aw10 = aw ^ 64, aw11 = aw ^ 72;
        u32x4 xf[2]; u32x2 wf[2][2][2];
        asm volatile(
          "ds_read_b128 %0, %10\n\t"
          "ds_read_b64_tr_b16 %2, %11\n\t"
          "ds_read_b64_tr_b16 %3, %11 offset:512\n\t"
          "ds_read_b64_tr_b16 %4, %12\n\t"
          "ds_read_b64_tr_b16 %5, %12 offset:512\n\t"
          "ds_read_b128 %1, %10 offset:1024\n\t"
          "ds_read_b64_tr_b16 %6, %13\n\t"
          "ds_read_b64_tr_b16 %7, %13 offset:512\n\t"
          "ds_read_b64_tr_b16 %8, %14\n\t"
          "ds_read_b64_tr_b16 %9, %14 offset:512\n\t"
          "s_waitcnt lgkmcnt(0)"
          : "=&v"(xf[0]), "=&v"(xf[1]), "=&v"(wf[0][0][0]), "=&v"(wf[0][0][1]), "=&v"(wf[0][1][0]), "=&v"(wf[0][1][1]),
            "=&v"(wf[1][0][0]), "=&v"(wf[1][0][1]), "=&v"(wf[1][1][0]), "=&v"(wf[1][1][1])
          : "v"(ax), "v"(aw00), "v"(aw01), "v"(aw10), "v"(aw11) : "memory");
#pragma unroll
        for (int i = 0; i < 4; ++i) {
          const int n_ = i >> 1, t_ = i & 1;
          const u32x4 a = {wf[n_][t_][0][0], wf[n_][t_][0][1], wf[n_][t_][1][0], wf[n_][t_][1][1]};
          acc[0][n_][t_] = __builtin_amdgcn_mfma_f32_16x16x32_bf16(__builtin_bit_cast(bf16x8, a), __builtin_bit_cast(bf16x8, xf[0]), acc[0][n_][t_], 0, 0, 0);
          acc[1][n_][t_] = __builtin_amdgcn_mfma_f32_16x16x32_bf16(__builtin_bit_cast(bf16x8, a), __builtin_bit_cast(bf16x8, xf[1]), acc[1][n_][t_], 0, 0, 0);
        }
      }
    }
    f32x4 acc0, acc1;
    {
      f32x4* mine = (f32x4*)(smem + LDS_W + wv * 4096) + lane;
#pragma unroll
      for (int i = 0; i < 8; ++i) mine[(i >> 2) * (WS_BYTES / 16) + (i & 3) * 64] = acc[i >> 2][(i >> 1) & 1][i & 1];
      asm volatile("s_waitcnt lgkmcnt(0)" ::: "memory");
      if (lane == 0) __hip_atomic_fetch_add(ctl + 3, 1u, __ATOMIC_RELAXED, __HIP_MEMORY_SCOPE_WORKGROUP);
      while (lds_ld(ctl + 3) < 4u) { __builtin_amdgcn_s_sleep(0); }
      asm volatile("" ::: "memory");
      const int cg = lane & 7, srcl = (wv & 1) * 8 + (lane >> 3) + 16 * (cg & 3);
      const f32x4* src = (const f32x4*)(smem + LDS_W + wm * WS_BYTES + (cg >> 2) * 2048) + srcl;
      acc0 = src[0]; acc1 = src[64];
#pragma unroll
      for (int q = 1; q < 4; ++q) { acc0 += src[q * 256]; acc1 += src[q * 256 + 64]; }
    }
    const bool bad = lds_ld(ctl + 4) != 0;
    const f32x4 lo = (lane & 1) ? acc1 : acc0, hi = (lane & 1) ? acc0 : acc1;
    u32x4 o;
    o[0] = pack_relu(lo[0], lo[1]); o[1] = pack_relu(lo[2], lo[3]);
    o[2] = pack_relu(hi[0], hi[1]); o[3] = pack_relu(hi[2], hi[3]);
    if (bad) { o[0] = o[1] = o[2] = o[3] = 0x7fc07fc0u; }
    const int grow = T.xsrc * MS + T.xt * 32 + wv * 8 + (lane >> 3);
    char* op = P.out[T.owner] + (size_t)grow * (NS * 2) + T.ct * 128 + (lane & 7) * 16;
    if (!wrem) {
      *(u32x4*)op = o;
    } else {
      st16_sys(op, o);
      WAITV(0);
      if (lane == 0) st4_sys((unsigned*)(P.ws[T.owner] + FLAG_OFF) + FY_W + ((rank >> 1) * 32 + T.mt) * 4 + wv, call);
    }
    if (wv == 0) STAMP(4);
  }
}

}

extern "C" void kernel_launch(int rank, int nranks, ncclComm_t comm,
                              hipStream_t stream,
                              void* const* d_in, const int* in_sizes, int n_in,
                              void* d_out, int out_size,
                              const int* in_dtypes, int out_dtype,
                              void* const* d_out_all,
                              void* const* d_ws_all, size_t ws_bytes) {
  if (nranks != NR || ws_bytes < WS_NEED || n_in < 2 || (unsigned)rank >= (unsigned)NR) {
    fprintf(stderr, "unsupported configuration\n");
    return;
  }
  Params p;
  p.x = (const char*)d_in[0];
  p.w = (const char*)d_in[1];
  for (int i = 0; i < NR; ++i) { p.out[i] = (char*)d_out_all[i]; p.ws[i] = (char*)d_ws_all[i]; }
  p.rank = rank;
  ag_gemm_kernel<<<NWG, NTHREADS, 0, stream>>>(p);
  HIP_CHECK(hipGetLastError());
}
```

```cpp
#include <hip/hip_runtime.h>
#include <rccl/rccl.h>
#include <cstdio>
#include <type_traits>

#define HIP_CHECK(expr) do { hipError_t _e = (expr); if (_e != hipSuccess) { \
    fprintf(stderr, "HIP error %s at %s:%d\n", hipGetErrorString(_e), __FILE__, __LINE__); \
    return; } } while (0)

#define MODE_AG 0
#define ACQ_MODE 2
#define PFD 5
#define PUSH_AFTER_FENCE 1
#define NPW 1
#define NP2 64
#define NPUSHWG 256
#define WT_OUT 1
#define PHASES 0
#define XMAP 1
#define PIPE 1
#define LD_AUX 0

namespace {

constexpr int NR = 8;
constexpr int KDIM = 2048;
constexpr int MS = 256, NS = 256;
constexpr int BM = 32, BN = 64, KC = 128, NC = KDIM / KC;
constexpr int NWG = 256;
constexpr int NWAVES = 5 + NPW;
constexpr int NTHREADS = NWAVES * 64;

constexpr int NMT = 32;
constexpr int PPC = 70;
constexpr int NPAIRS = PPC * NC;
constexpr int NP = NPUSHWG + (NPW - 1) * NP2;
static_assert(NPAIRS >= 2 * NP, "two static units per pusher wave");

constexpr size_t XT_BYTES = (size_t)BM * KDIM * 2;
constexpr size_t WT_BYTES = (size_t)BN * KDIM * 2;
constexpr size_t XSTG_BYTES = (size_t)NMT * XT_BYTES;
constexpr size_t WSTG_BYTES = MODE_AG ? 0 : 4 * WT_BYTES;
constexpr size_t PAR_BYTES = XSTG_BYTES + WSTG_BYTES;
constexpr size_t FLAG_OFF = 16u << 20;
static_assert(2 * PAR_BYTES <= FLAG_OFF, "staging overflows");
constexpr size_t WS_NEED = FLAG_OFF + (256u << 10);
constexpr int FX_W = 0;
constexpr int FW_W = 2048;
constexpr int FY_W = 4096;
constexpr int CNT_W = 8192;
constexpr int TMO_W = 8192 + 1024;
constexpr int TICKET_W = 8192 + 2048;
constexpr int DBG_W = 12288;

constexpr unsigned long long DEADLINE = 50000000ull;

constexpr int XS_BYTES = BM * KC * 2;
constexpr int WS_BYTES = BN * KC * 2;
constexpr int NSLOT = 6;
constexpr int LDS_X = 0;
constexpr int LDS_W = NSLOT * XS_BYTES;
constexpr int LDS_CTL = LDS_W + NSLOT * WS_BYTES;
constexpr int LDS_BYTES = LDS_CTL + 64;

typedef __attribute__((ext_vector_type(4))) unsigned u32x4;
typedef __attribute__((ext_vector_type(2))) unsigned u32x2;
typedef __attribute__((ext_vector_type(8))) short bf16x8;
typedef __attribute__((ext_vector_type(4))) float f32x4;
typedef __attribute__((ext_vector_type(2))) float f32x2;
typedef __attribute__((ext_vector_type(2))) __bf16 bf16x2;
typedef __attribute__((address_space(3))) unsigned lds_u32;

struct Params {
  const char* x;
  const char* w;
  char* out[NR];
  char* ws[NR];
  int rank;
};

__device__ __forceinline__ void st16_sys(void* p, u32x4 v) {
  asm volatile("global_store_dwordx4 %0, %1, off sc0 sc1\n\ts_nop 1" :: "v"(p), "v"(v) : "memory");
}
__device__ __forceinline__ void st4_sys(void* p, unsigned v) {
  asm volatile("global_store_dword %0, %1, off sc0 sc1" :: "v"(p), "v"(v) : "memory");
}
__device__ __forceinline__ unsigned ld4_sys(const void* p) {
  return __hip_atomic_load((const unsigned*)p, __ATOMIC_RELAXED, __HIP_MEMORY_SCOPE_SYSTEM);
}
__device__ __forceinline__ unsigned lds_ld(unsigned* p) {
  return __hip_atomic_load(p, __ATOMIC_RELAXED, __HIP_MEMORY_SCOPE_WORKGROUP);
}
__device__ __forceinline__ void lds_st(unsigned* p, unsigned v) {
  __hip_atomic_store(p, v, __ATOMIC_RELAXED, __HIP_MEMORY_SCOPE_WORKGROUP);
}
#define WAITV(n) asm volatile("s_waitcnt vmcnt(" #n ")" ::: "memory")

template <int AUX>
__device__ __forceinline__ void dma16(const char* g, unsigned l) {
  if constexpr (AUX == 0)
    asm volatile("s_mov_b32 m0, %1\n\ts_nop 0\n\tglobal_load_lds_dwordx4 %0, off" :: "v"(g), "s"(l) : "memory");
  else if constexpr (AUX == 16)
    asm volatile("s_mov_b32 m0, %1\n\ts_nop 0\n\tglobal_load_lds_dwordx4 %0, off sc1" :: "v"(g), "s"(l) : "memory");
  else if constexpr (AUX == 17)
    asm volatile("s_mov_b32 m0, %1\n\ts_nop 0\n\tglobal_load_lds_dwordx4 %0, off sc0 sc1" :: "v"(g), "s"(l) : "memory");
  else
    asm volatile("s_mov_b32 m0, %1\n\ts_nop 0\n\tglobal_load_lds_dwordx4 %0, off nt" :: "v"(g), "s"(l) : "memory");
}

__device__ __forceinline__ unsigned pack_relu(float a, float b) {
  a = a < 0.f ? 0.f : a;
  b = b < 0.f ? 0.f : b;
  f32x2 f = {a, b};
  bf16x2 h = __builtin_convertvector(f, bf16x2);
  return __builtin_bit_cast(unsigned, h);
}

#define STAMP(idx) do {} while (0)

struct Tile { int mt, xsrc, xt, owner, ct, wslot; };

__device__ __forceinline__ void get_tile(int rank, int b, Tile& T) {
  const int xcd = b & 7, s = b >> 3;
  const int nt = (xcd >> 2) * 4 + (s >> 3);
  const int mt = (xcd & 3) * 8 + (s & 7);
  const int par_me = rank & 1;
  T.wslot = nt - 4;
  T.owner = nt < 4 ? rank : (1 - par_me) + 2 * (nt - 4);
  T.ct = nt < 4 ? nt : (rank >> 1);
  T.mt = mt;
  if (mt < 20) { T.xsrc = par_me + 2 * (mt / 5); T.xt = mt % 5; }
  else { T.xsrc = (1 - par_me) + 2 * ((mt - 20) / 3); T.xt = 5 + (mt - 20) % 3; }
}

__global__ void __launch_bounds__(NTHREADS, 1) ag_gemm_kernel(Params P) {
  __shared__ __attribute__((aligned(1024))) char smem[LDS_BYTES];
  unsigned* ctl = (unsigned*)(smem + LDS_CTL);
  const int tid = threadIdx.x, lane = tid & 63;
  const int wv = __builtin_amdgcn_readfirstlane(tid >> 6);
  const int b = blockIdx.x;
  const int rank = P.rank;
  char* myws = P.ws[rank];
  unsigned* myf = (unsigned*)(myws + FLAG_OFF);
  unsigned* tmo = myf + TMO_W;

  if (wv == 0) STAMP(0);

  if (wv >= 5) {
    const int g = wv == 5 ? b : NPUSHWG + b;
    __builtin_amdgcn_s_barrier();
    if (b >= (wv == 5 ? NPUSHWG : NP2)) return;
    unsigned call = ld4_sys(myf + CNT_W + NWG + g) + 1;
    const int r = lane >> 3;
    const int r16 = lane >> 2;
    const unsigned xl = r16 * (KDIM * 2) + ((lane & 3) ^ ((r16 >> 2) & 3)) * 16;
    const unsigned wl0 = r * (NS * 2) + ((lane & 7) ^ ((r >> 1) & 1)) * 16;
    const unsigned wl1 = (r + 8) * (NS * 2) + ((lane & 7) ^ (((r >> 1) & 1) | 4)) * 16;
    struct Unit { u32x4 v[4]; char* dp; unsigned* fp; };
    auto fetch = [&](int i, Unit& U) {
      const int c = i / PPC, q = i % PPC;
      const int d = 1 + q % 7, u = q / 7;
      const int dest = (rank + d) & 7;
      char* dws = P.ws[dest];
      bool isw = false; int xu = u;
      if (d & 1) { if (u < 4) isw = true; else xu = 10 + (u - 4); }
      const int t = xu >> 1, h = xu & 1;
      const int dmt = t < 5 ? (rank >> 1) * 5 + t : 20 + (rank >> 1) * 3 + (t - 5);
      const char* sb = isw ? P.w + (size_t)(c * KC + u * 32) * (NS * 2) + (dest >> 1) * 128
                           : P.x + (size_t)(t * 32) * (KDIM * 2) + c * (KC * 2) + h * 128;
      const unsigned o0 = isw ? wl0 : xl, o1 = isw ? wl1 : xl + 16 * KDIM * 2;
      const unsigned o2 = isw ? wl0 + 16 * NS * 2 : xl + 64, o3 = isw ? wl1 + 16 * NS * 2 : xl + 64 + 16 * KDIM * 2;
      U.v[0] = *(const u32x4*)(sb + o0);
      U.v[1] = *(const u32x4*)(sb + o1);
      U.v[2] = *(const u32x4*)(sb + o2);
      U.v[3] = *(const u32x4*)(sb + o3);
      U.dp = dws + (isw ? (unsigned)(XSTG_BYTES + (rank >> 1) * WT_BYTES) + c * WS_BYTES + u * 4096
                        : (unsigned)(dmt * XT_BYTES) + c * XS_BYTES + h * 4096) + lane * 16;
      U.fp = (unsigned*)(dws + FLAG_OFF) + (isw ? FW_W + ((rank >> 1) * NC + c) * 4 + u : FX_W + (dmt * NC + c) * 2 + h);
    };
    Unit UA, UB;
    fetch(g, UA);
    fetch(g + NP, UB);
    if (lane == 0) st4_sys(myf + CNT_W + NWG + g, call);
    const size_t poff = (size_t)(call & 1) * PAR_BYTES;
    const unsigned tbase = (call - 1) * (unsigned)(NPAIRS - NP) - 2u * NP;
    STAMP(6);
    int nxt = g + NP;
    auto step = [&](Unit& U) -> bool {
      char* dp = U.dp + poff;
#pragma unroll
      for (int ks = 0; ks < 4; ++ks) st16_sys(dp + ks * 1024, U.v[ks]);
      unsigned tk = 0;
      const bool more = nxt < NPAIRS;
      if (more && lane == 0) tk = __hip_atomic_fetch_add(myf + TICKET_W, 1u, __ATOMIC_RELAXED, __HIP_MEMORY_SCOPE_SYSTEM);
      WAITV(0);
      if (lane == 0) st4_sys(U.fp, call);
      if (!more) return false;
      nxt = (int)(__builtin_amdgcn_readfirstlane(tk) - tbase);
      if (nxt < NPAIRS) fetch(nxt, U);
      return true;
    };
    while (true) {
      if (!step(UA)) break;
      if (!step(UB)) break;
    }
    STAMP(1);
    return;
  }

  if (wv == 4) {
    unsigned call = 0;
    if (lane < 8) lds_st(ctl + lane, 0u);
    asm volatile("s_waitcnt lgkmcnt(0)" ::: "memory");
    __builtin_amdgcn_s_barrier();
    if (lane == 0) {
      call = ld4_sys(myf + CNT_W + b) + 1;
      st4_sys(myf + CNT_W + b, call);
    }
    call = __builtin_amdgcn_readfirstlane(call);
    __builtin_amdgcn_fence(__ATOMIC_ACQUIRE, "");
    asm volatile("s_waitcnt vmcnt(0)" ::: "memory");
    if (lane == 0) lds_st(ctl + 0, call);
    const unsigned long long t0 = __builtin_amdgcn_s_memrealtime();
    bool bad = false;
    {
      Tile T;
      get_tile(rank, b, T);
      const bool xrem = T.xsrc != rank, wrem = T.owner != rank;
      int rdy = 0;
      if (!xrem && !wrem) { rdy = NC; if (lane == 0) lds_st(ctl + 1, (unsigned)NC); }
      unsigned it = 0;
      while (rdy < NC) {
        unsigned f = call;
        const int cx = rdy + (lane >> 1), cw = rdy + ((lane - 8) >> 2);
        if (lane < 8) { if (xrem && cx < NC) f = ld4_sys(myf + FX_W + T.mt * 32 + rdy * 2 + lane); }
        else if (lane < 24) { if (wrem && cw < NC) f = ld4_sys(myf + FW_W + T.wslot * 64 + rdy * 4 + (lane - 8)); }
        unsigned tm = 0;
        if ((++it & 15) == 0) tm = __hip_atomic_load(tmo, __ATOMIC_RELAXED, __HIP_MEMORY_SCOPE_SYSTEM);
        const unsigned bl = (unsigned)__ballot(f >= call);
        unsigned bx = bl & 0xffu, bw = (bl >> 8) & 0xffffu;
        bx &= bx >> 1; bx &= 0x55u;
        bw &= bw >> 1; bw &= bw >> 2; bw &= 0x1111u;
        const int nrx = __builtin_ctz((~bx & 0x55u) | 0x100u) >> 1;
        const int nrw = __builtin_ctz((~bw & 0x1111u) | 0x10000u) >> 2;
        int nr = rdy + (nrx < nrw ? nrx : nrw);
        nr = nr < NC ? nr : NC;
        if (tm == call || ((it & 15) == 0 && __builtin_amdgcn_s_memrealtime() - t0 > DEADLINE)) {
          if (lane == 0) { __hip_atomic_store(tmo, call, __ATOMIC_RELAXED, __HIP_MEMORY_SCOPE_SYSTEM); lds_st(ctl + 4, 1u); }
          bad = true; nr = NC;
        }
        if (nr > rdy) {
          if (lane == 0) lds_st(ctl + 1, (unsigned)nr);
          rdy = nr;
        } else {
          __builtin_amdgcn_s_sleep(1);
        }
      }
    }
    STAMP(2);
    if (b < 8) {
      bool done = false;
      while (!done) {
        const unsigned fy = ld4_sys(myf + FY_W + b * 64 + lane);
        const unsigned tm = __hip_atomic_load(tmo, __ATOMIC_RELAXED, __HIP_MEMORY_SCOPE_SYSTEM);
        done = __all(fy >= call);
        if (!done) {
          if (tm == call || __builtin_amdgcn_s_memrealtime() - t0 > DEADLINE) {
            if (lane == 0) __hip_atomic_store(tmo, call, __ATOMIC_RELAXED, __HIP_MEMORY_SCOPE_SYSTEM);
            bad = true; done = true;
          } else __builtin_amdgcn_s_sleep(1);
        }
      }
      if (bad && lane == 0) {
        ((unsigned short*)P.out[rank])[(size_t)(rank * MS) * NS + b] = 0x7fc0;
      }
      STAMP(3);
    }
    return;
  }

  const int wm = wv >> 1, wn = wv & 1;
  const unsigned lbase = (unsigned)(size_t)(__attribute__((address_space(3))) char*)smem;
  const int r8 = lane >> 3;
  __builtin_amdgcn_s_barrier();
  asm volatile("" ::: "memory");
  unsigned call = lds_ld(ctl + 0);
  while (call == 0) { __builtin_amdgcn_s_sleep(0); call = lds_ld(ctl + 0); }
  asm volatile("" ::: "memory");
  const char* sbase = myws + (size_t)(call & 1) * PAR_BYTES;

  const int m16 = lane & 15, g4 = lane >> 4, q4 = (lane & 15) >> 2, p4 = lane & 3;
  const unsigned xa0 = wv * 2048 + m16 * 64 + ((g4 ^ ((m16 >> 2) & 3)) * 16);
  const unsigned fwl = ((q4 >> 1) & 1) | ((g4 & 1) << 2);
  const unsigned wa_0 = wv * 4096 + g4 * 1024 + q4 * 128 + ((p4 ^ fwl) * 16) + ((p4 & 1) * 8);

  {
    Tile T;
    get_tile(rank, b, T);
    const bool xrem = T.xsrc != rank, wrem = T.owner != rank;
    const char* xg; unsigned xstride_c, xstride_ks;
    if (xrem) {
      xg = sbase + (size_t)T.mt * XT_BYTES + wv * 2048 + lane * 16; xstride_c = XS_BYTES; xstride_ks = 1024;
    } else {
      const int r16 = lane >> 2;
      const int c4 = (lane & 3) ^ ((r16 >> 2) & 3);
      xg = P.x + (size_t)(T.xt * 32 + r16) * (KDIM * 2) + wv * 64 + c4 * 16; xstride_c = KC * 2; xstride_ks = 16 * KDIM * 2;
    }
    const char* wg0; const char* wg1; unsigned wstride_c, wstride_k2;
    if (wrem) {
      wg0 = sbase + XSTG_BYTES + (size_t)T.wslot * WT_BYTES + wv * 4096 + lane * 16; wg1 = wg0 + 1024;
      wstride_c = WS_BYTES; wstride_k2 = 2048;
    } else {
      const int f0 = ((r8 >> 1) & 1), f1 = f0 | 4;
      wg0 = P.w + (size_t)(wv * 32 + r8) * (NS * 2) + T.ct * 128 + (((lane & 7) ^ f0) * 16);
      wg1 = P.w + (size_t)(wv * 32 + 8 + r8) * (NS * 2) + T.ct * 128 + (((lane & 7) ^ f1) * 16);
      wstride_c = KC * NS * 2; wstride_k2 = 16 * NS * 2;
    }
    f32x4 acc[2][2][2];
#pragma unroll
    for (int i = 0; i < 8; ++i) acc[i >> 2][(i >> 1) & 1][i & 1] = f32x4{0.f, 0.f, 0.f, 0.f};
    constexpr int cbase = 0;
    int issued = 0;
    for (int c = 0; c < NC; ++c) {
      int rdy = (int)lds_ld(ctl + 1) - cbase;
      while (issued <= c && rdy <= issued) { __builtin_amdgcn_s_sleep(0); rdy = (int)lds_ld(ctl + 1) - cbase; }
      asm volatile("" ::: "memory");
      int lim = c + PFD;
      lim = lim < rdy ? lim : rdy;
      lim = lim < NC ? lim : NC;
      while (issued < lim) {
        const int sl = (cbase + issued) % NSLOT;
        const unsigned lw = lbase + LDS_W + sl * WS_BYTES + wv * 4096;
        const unsigned lx = lbase + LDS_X + sl * XS_BYTES + wv * 2048;
        const char* w0 = wg0 + (size_t)issued * wstride_c;
        const char* w1 = wg1 + (size_t)issued * wstride_c;
        const char* x0 = xg + (size_t)issued * xstride_c;
        {
        dma16<LD_AUX>(w0, lw);
        dma16<LD_AUX>(w1, lw + 1024);
        dma16<LD_AUX>(w0 + wstride_k2, lw + 2048);
        dma16<LD_AUX>(w1 + wstride_k2, lw + 3072);
        dma16<LD_AUX>(x0, lx);
        dma16<LD_AUX>(x0 + xstride_ks, lx + 1024);
        }
        ++issued;
      }
      switch (issued - c - 1) {
        case 0: WAITV(0); break;
        case 1: WAITV(6); break;
        case 2: WAITV(12); break;
        case 3: WAITV(18); break;
        default: WAITV(24); break;
      }
      {
        const int sl = c % NSLOT;
        const unsigned ax = lbase + LDS_X + sl * XS_BYTES + xa0;
        const unsigned aw = lbase + LDS_W + sl * WS_BYTES + wa_0;
        const unsigned aw00 = aw, aw01 = aw ^ 8, aw10 = aw ^ 64, aw11 = aw ^ 72;
        u32x4 xf[2]; u32x2 wf[2][2][2];
        asm volatile(
          "ds_read_b128 %0, %10\n\t"
          "ds_read_b64_tr_b16 %2, %11\n\t"
          "ds_read_b64_tr_b16 %3, %11 offset:512\n\t"
          "ds_read_b64_tr_b16 %4, %12\n\t"
          "ds_read_b64_tr_b16 %5, %12 offset:512\n\t"
          "ds_read_b128 %1, %10 offset:1024\n\t"
          "ds_read_b64_tr_b16 %6, %13\n\t"
          "ds_read_b64_tr_b16 %7, %13 offset:512\n\t"
          "ds_read_b64_tr_b16 %8, %14\n\t"
          "ds_read_b64_tr_b16 %9, %14 offset:512\n\t"
          "s_waitcnt lgkmcnt(0)"
          : "=&v"(xf[0]), "=&v"(xf[1]), "=&v"(wf[0][0][0]), "=&v"(wf[0][0][1]), "=&v"(wf[0][1][0]), "=&v"(wf[0][1][1]),
            "=&v"(wf[1][0][0]), "=&v"(wf[1][0][1]), "=&v"(wf[1][1][0]), "=&v"(wf[1][1][1])
          : "v"(ax), "v"(aw00), "v"(aw01), "v"(aw10), "v"(aw11) : "memory");
#pragma unroll
        for (int i = 0; i < 4; ++i) {
          const int n_ = i >> 1, t_ = i & 1;
          const u32x4 a = {wf[n_][t_][0][0], wf[n_][t_][0][1], wf[n_][t_][1][0], wf[n_][t_][1][1]};
          acc[0][n_][t_] = __builtin_amdgcn_mfma_f32_16x16x32_bf16(__builtin_bit_cast(bf16x8, a), __builtin_bit_cast(bf16x8, xf[0]), acc[0][n_][t_], 0, 0, 0);
          acc[1][n_][t_] = __builtin_amdgcn_mfma_f32_16x16x32_bf16(__builtin_bit_cast(bf16x8, a), __builtin_bit_cast(bf16x8, xf[1]), acc[1][n_][t_], 0, 0, 0);
        }
      }
    }
    f32x4 acc0, acc1;
    {
      f32x4* mine = (f32x4*)(smem + LDS_W + wv * 4096) + lane;
#pragma unroll
      for (int i = 0; i < 8; ++i) mine[(i >> 2) * (WS_BYTES / 16) + (i & 3) * 64] = acc[i >> 2][(i >> 1) & 1][i & 1];
      asm volatile("s_waitcnt lgkmcnt(0)" ::: "memory");
      if (lane == 0) __hip_atomic_fetch_add(ctl + 3, 1u, __ATOMIC_RELAXED, __HIP_MEMORY_SCOPE_WORKGROUP);
      while (lds_ld(ctl + 3) < 4u) { __builtin_amdgcn_s_sleep(0); }
      asm volatile("" ::: "memory");
      const int cg = lane & 7, srcl = (wv & 1) * 8 + (lane >> 3) + 16 * (cg & 3);
      const f32x4* src = (const f32x4*)(smem + LDS_W + wm * WS_BYTES + (cg >> 2) * 2048) + srcl;
      acc0 = src[0]; acc1 = src[64];
#pragma unroll
      for (int q = 1; q < 4; ++q) { acc0 += src[q * 256]; acc1 += src[q * 256 + 64]; }
    }
    const bool bad = lds_ld(ctl + 4) != 0;
    const f32x4 lo = (lane & 1) ? acc1 : acc0, hi = (lane & 1) ? acc0 : acc1;
    u32x4 o;
    o[0] = pack_relu(lo[0], lo[1]); o[1] = pack_relu(lo[2], lo[3]);
    o[2] = pack_relu(hi[0], hi[1]); o[3] = pack_relu(hi[2], hi[3]);
    if (bad) { o[0] = o[1] = o[2] = o[3] = 0x7fc07fc0u; }
    const int grow = T.xsrc * MS + T.xt * 32 + wv * 8 + (lane >> 3);
    char* op = P.out[T.owner] + (size_t)grow * (NS * 2) + T.ct * 128 + (lane & 7) * 16;
    if (!wrem) {
      st16_sys(op, o);
    } else {
      st16_sys(op, o);
      WAITV(0);
      if (lane == 0) st4_sys((unsigned*)(P.ws[T.owner] + FLAG_OFF) + FY_W + ((rank >> 1) * 32 + T.mt) * 4 + wv, call);
    }
    if (wv == 0) STAMP(4);
  }
}

}

extern "C" void kernel_launch(int rank, int nranks, ncclComm_t comm,
                              hipStream_t stream,
                              void* const* d_in, const int* in_sizes, int n_in,
                              void* d_out, int out_size,
                              const int* in_dtypes, int out_dtype,
                              void* const* d_out_all,
                              void* const* d_ws_all, size_t ws_bytes) {
  if (nranks != NR || ws_bytes < WS_NEED || n_in < 2 || (unsigned)rank >= (unsigned)NR) {
    fprintf(stderr, "unsupported configuration\n");
    return;
  }
  Params p;
  p.x = (const char*)d_in[0];
  p.w = (const char*)d_in[1];
  for (int i = 0; i < NR; ++i) { p.out[i] = (char*)d_out_all[i]; p.ws[i] = (char*)d_ws_all[i]; }
  p.rank = rank;
  ag_gemm_kernel<<<NWG, NTHREADS, 0, stream>>>(p);
  HIP_CHECK(hipGetLastError());
}
```

```cpp
#include <hip/hip_runtime.h>
#include <rccl/rccl.h>
#include <cstdio>
#include <type_traits>

#define HIP_CHECK(expr) do { hipError_t _e = (expr); if (_e != hipSuccess) { \
    fprintf(stderr, "HIP error %s at %s:%d\n", hipGetErrorString(_e), __FILE__, __LINE__); \
    return; } } while (0)

#define MODE_AG 0
#define ACQ_MODE 2
#define PFD 5
#define PUSH_AFTER_FENCE 1
#define NPW 1
#define NP2 64
#define NPUSHWG 256
#define WT_OUT 1
#define PHASES 0
#define XMAP 1
#define PIPE 1
#define WARM 1
#define AFF 1
#define WARM_END 0
#define WARMP 0
#define RAWP 1
#define SUBW 2
#define TAILX 0
#define FASTP 0
#define PS_FAST 12
#define PS_SLOW 24
#define FASTC 10
#define WARMC 1
#define LD_AUX 0

namespace {

constexpr int NR = 8;
constexpr int KDIM = 2048;
constexpr int MS = 256, NS = 256;
constexpr int BM = 32, BN = 64, KC = 128, NC = KDIM / KC;
constexpr int NWG = 256;
constexpr int NWAVES = 5 + NPW;
constexpr int NTHREADS = NWAVES * 64;

constexpr int NMT = 32;
constexpr int PPC = 70;
constexpr int NPAIRS = PPC * NC;
constexpr int NP = NPUSHWG + (NPW - 1) * NP2;
static_assert(NPAIRS >= 2 * NP, "two static units per pusher wave");

constexpr size_t XT_BYTES = (size_t)BM * KDIM * 2;
constexpr size_t WT_BYTES = (size_t)BN * KDIM * 2;
constexpr size_t XSTG_BYTES = (size_t)NMT * XT_BYTES;
constexpr size_t WSTG_BYTES = MODE_AG ? 0 : 4 * WT_BYTES;
constexpr size_t PAR_BYTES = XSTG_BYTES + WSTG_BYTES;
constexpr size_t FLAG_OFF = 16u << 20;
static_assert(2 * PAR_BYTES <= FLAG_OFF, "staging overflows");
constexpr size_t WS_NEED = FLAG_OFF + (256u << 10);
constexpr int FX_W = 0;
constexpr int FW_W = 2048;
constexpr int FY_W = 4096;
constexpr int CNT_W = 8192;
constexpr int TMO_W = 8192 + 1024;
constexpr int TICKET_W = 8192 + 2048;
constexpr int DBG_W = 12288;

constexpr unsigned long long DEADLINE = 50000000ull;

constexpr int XS_BYTES = BM * KC * 2;
constexpr int WS_BYTES = BN * KC * 2;
constexpr int NSLOT = 6;
constexpr int LDS_X = 0;
constexpr int LDS_W = NSLOT * XS_BYTES;
constexpr int LDS_CTL = LDS_W + NSLOT * WS_BYTES;
constexpr int LDS_BYTES = LDS_CTL + 64;

typedef __attribute__((ext_vector_type(4))) unsigned u32x4;
typedef __attribute__((ext_vector_type(2))) unsigned u32x2;
typedef __attribute__((ext_vector_type(8))) short bf16x8;
typedef __attribute__((ext_vector_type(4))) float f32x4;
typedef __attribute__((ext_vector_type(2))) float f32x2;
typedef __attribute__((ext_vector_type(2))) __bf16 bf16x2;
typedef __attribute__((address_space(3))) unsigned lds_u32;

struct Params {
  const char* x;
  const char* w;
  char* out[NR];
  char* ws[NR];
  int rank;
};

__device__ __forceinline__ void st16_sys(void* p, u32x4 v) {
  asm volatile("global_store_dwordx4 %0, %1, off sc0 sc1\n\ts_nop 1" :: "v"(p), "v"(v) : "memory");
}
__device__ __forceinline__ void st4_sys(void* p, unsigned v) {
  asm volatile("global_store_dword %0, %1, off sc0 sc1" :: "v"(p), "v"(v) : "memory");
}
__device__ __forceinline__ unsigned ld4_sys(const void* p) {
  return __hip_atomic_load((const unsigned*)p, __ATOMIC_RELAXED, __HIP_MEMORY_SCOPE_SYSTEM);
}
__device__ __forceinline__ unsigned lds_ld(unsigned* p) {
  return __hip_atomic_load(p, __ATOMIC_RELAXED, __HIP_MEMORY_SCOPE_WORKGROUP);
}
__device__ __forceinline__ void lds_st(unsigned* p, unsigned v) {
  __hip_atomic_store(p, v, __ATOMIC_RELAXED, __HIP_MEMORY_SCOPE_WORKGROUP);
}
#define WAITV(n) asm volatile("s_waitcnt vmcnt(" #n ")" ::: "memory")

template <int AUX>
__device__ __forceinline__ void dma16(const char* g, unsigned l) {
  if constexpr (AUX == 0)
    asm volatile("s_mov_b32 m0, %1\n\ts_nop 0\n\tglobal_load_lds_dwordx4 %0, off" :: "v"(g), "s"(l) : "memory");
  else if constexpr (AUX == 16)
    asm volatile("s_mov_b32 m0, %1\n\ts_nop 0\n\tglobal_load_lds_dwordx4 %0, off sc1" :: "v"(g), "s"(l) : "memory");
  else if constexpr (AUX == 17)
    asm volatile("s_mov_b32 m0, %1\n\ts_nop 0\n\tglobal_load_lds_dwordx4 %0, off sc0 sc1" :: "v"(g), "s"(l) : "memory");
  else
    asm volatile("s_mov_b32 m0, %1\n\ts_nop 0\n\tglobal_load_lds_dwordx4 %0, off nt" :: "v"(g), "s"(l) : "memory");
}

__device__ __forceinline__ unsigned pack_relu(float a, float b) {
  a = a < 0.f ? 0.f : a;
  b = b < 0.f ? 0.f : b;
  f32x2 f = {a, b};
  bf16x2 h = __builtin_convertvector(f, bf16x2);
  return __builtin_bit_cast(unsigned, h);
}

#define STAMP(idx) do {} while (0)

struct Tile { int mt, xsrc, xt, owner, ct, wslot; };

__device__ __forceinline__ void get_tile(int rank, int b, Tile& T) {
  const int xcd = b & 7, s = b >> 3;
  const int nt = (xcd >> 2) * 4 + (s >> 3);
  const int mt = (xcd & 3) * 8 + (s & 7);
  const int par_me = rank & 1;
  T.wslot = nt - 4;
  T.owner = nt < 4 ? rank : (1 - par_me) + 2 * (nt - 4);
  T.ct = nt < 4 ? nt : (rank >> 1);
  T.mt = mt;
  if (mt < 20) { T.xsrc = par_me + 2 * (mt / 5); T.xt = mt % 5; }
  else { T.xsrc = (1 - par_me) + 2 * ((mt - 20) / 3); T.xt = 5 + (mt - 20) % 3; }
}

__global__ void __launch_bounds__(NTHREADS, 1) ag_gemm_kernel(Params P) {
  __shared__ __attribute__((aligned(1024))) char smem[LDS_BYTES];
  unsigned* ctl = (unsigned*)(smem + LDS_CTL);
  const int tid = threadIdx.x, lane = tid & 63;
  const int wv = __builtin_amdgcn_readfirstlane(tid >> 6);
  const int b = blockIdx.x;
  const int rank = P.rank;
  char* myws = P.ws[rank];
  unsigned* myf = (unsigned*)(myws + FLAG_OFF);
  unsigned* tmo = myf + TMO_W;

  if (wv == 0) STAMP(0);
  const char* code_pc; int code_len;
  asm volatile("agk_code_begin:\n\ts_getpc_b64 %0\n\ts_mov_b32 %1, agk_code_end0-agk_code_begin-8\n\t"
               "s_max_i32 %1, %1, agk_code_end1-agk_code_begin-8\n\ts_max_i32 %1, %1, agk_code_end2-agk_code_begin-8"
               : "=s"(code_pc), "=s"(code_len) :: "scc");

  if (wv >= 5) {
    const int g = wv == 5 ? b : NPUSHWG + b;
    __builtin_amdgcn_s_barrier();
    if (b >= (wv == 5 ? NPUSHWG : NP2)) return;
    unsigned call;
    asm volatile("global_load_dword %0, %1, off sc0 sc1" : "=v"(call) : "v"(myf + CNT_W + NWG + g) : "memory");
    const int r = lane >> 3;
    const int r16 = lane >> 2;
    const unsigned xl = r16 * (KDIM * 2) + ((lane & 3) ^ ((r16 >> 2) & 3)) * 16;
    const unsigned wl0 = r * (NS * 2) + ((lane & 7) ^ ((r >> 1) & 1)) * 16;
    const unsigned wl1 = (r + 8) * (NS * 2) + ((lane & 7) ^ (((r >> 1) & 1) | 4)) * 16;
    struct Unit { u32x4 v[4]; char* dp; unsigned* fp; int idx; };
    auto fetch = [&](int i, Unit& U) {
      const int c = i / (PPC / 7);
      const int d = 1 + g % 7, u = i % (PPC / 7);
      const int dest = (rank + d) & 7;
      char* dws = P.ws[dest];
      bool isw = false; int xu = u;
      if (d & 1) { if (u < 4) isw = true; else xu = 10 + (u - 4); }
      const int t = xu >> 1, h = xu & 1;
      const int dmt = t < 5 ? (rank >> 1) * 5 + t : 20 + (rank >> 1) * 3 + (t - 5);
      const char* sb = isw ? P.w + (size_t)(c * KC + u * 32) * (NS * 2) + (dest >> 1) * 128
                           : P.x + (size_t)(t * 32) * (KDIM * 2) + c * (KC * 2) + h * 128;
      const unsigned o0 = isw ? wl0 : xl, o1 = isw ? wl1 : xl + 16 * KDIM * 2;
      const unsigned o2 = isw ? wl0 + 16 * NS * 2 : xl + 64, o3 = isw ? wl1 + 16 * NS * 2 : xl + 64 + 16 * KDIM * 2;
      asm volatile("global_load_dwordx4 %0, %1, off" : "=v"(U.v[0]) : "v"(sb + o0) : "memory");
      asm volatile("global_load_dwordx4 %0, %1, off" : "=v"(U.v[1]) : "v"(sb + o1) : "memory");
      asm volatile("global_load_dwordx4 %0, %1, off" : "=v"(U.v[2]) : "v"(sb + o2) : "memory");
      asm volatile("global_load_dwordx4 %0, %1, off" : "=v"(U.v[3]) : "v"(sb + o3) : "memory");
      U.idx = i;
      U.dp = dws + (isw ? (unsigned)(XSTG_BYTES + (rank >> 1) * WT_BYTES) + c * WS_BYTES + u * 4096
                        : (unsigned)(dmt * XT_BYTES) + c * XS_BYTES + h * 4096) + lane * 16;
      U.fp = (unsigned*)(dws + FLAG_OFF) + (isw ? FW_W + ((rank >> 1) * NC + c) * 4 + u : FX_W + (dmt * NC + c) * 2 + h);
    };
    Unit UA, UB;
    const int g0 = g / 7, NPL = NP / 7 + (g % 7 < NP % 7 ? 1 : 0), NPAIRSL = NPAIRS / 7;
    unsigned* ticket = myf + TICKET_W + (g % 7) * 64;
    fetch(g0, UA);
    fetch(g0 + NPL, UB);
    WAITV(4);
    asm volatile("" : "+v"(call));
    call = __builtin_amdgcn_readfirstlane(call) + 1;
    if (lane == 0) st4_sys(myf + CNT_W + NWG + g, call);
    const size_t poff = (size_t)(call & 1) * PAR_BYTES;
    const unsigned tbase = (call - 1) * (unsigned)(NPAIRSL - NPL) - 2u * NPL;
    STAMP(6);
    int nxt = g0 + NPL;
    auto step = [&](Unit& U) -> bool {
      char* dp = U.dp + poff;
#pragma unroll
      for (int ks = 0; ks < 4; ++ks) {
        st16_sys(dp + ks * 1024, U.v[ks]);
        if ((ks == 1 || ks == 2) && (TAILX == 0 || U.idx < NPAIRSL - TAILX)) WAITV(1);
      }
      unsigned tk = 0;
      const bool more = nxt < NPAIRSL;
      if (more && lane == 0) tk = __hip_atomic_fetch_add(ticket, 1u, __ATOMIC_RELAXED, __HIP_MEMORY_SCOPE_SYSTEM);
      WAITV(0);
      if (lane == 0) st4_sys(U.fp, call);
      if (!more) return false;
      nxt = (int)(__builtin_amdgcn_readfirstlane(tk) - tbase);
      if (nxt < NPAIRSL) fetch(nxt, U);
      return true;
    };
    while (true) {
      if (!step(UA)) break;
      if (!step(UB)) break;
    }
    STAMP(1);
    asm volatile("agk_code_end1:");
    return;
  }

  if (wv == 4) {
    unsigned call = 0;
    if (lane < 8) lds_st(ctl + lane, 0u);
    asm volatile("s_waitcnt lgkmcnt(0)" ::: "memory");
    __builtin_amdgcn_s_barrier();
    unsigned warmv = 0;
    {
      Tile T;
      get_tile(rank, b, T);
      if (lane == 1) warmv = ld4_sys(P.out[T.owner] + (size_t)(T.xsrc * MS + T.xt * 32) * (NS * 2) + T.ct * 128);
      if (lane == 2) warmv = ld4_sys((unsigned*)(P.ws[T.owner] + FLAG_OFF) + FY_W);
    }
    if (lane == 0) {
      call = ld4_sys(myf + CNT_W + b) + 1;
      st4_sys(myf + CNT_W + b, call);
    }
    call = __builtin_amdgcn_readfirstlane(call);
    __builtin_amdgcn_fence(__ATOMIC_ACQUIRE, "");
    asm volatile("s_waitcnt vmcnt(0)" ::: "memory");
    if (lane == 0) lds_st(ctl + 0, call);
    asm volatile("" :: "v"(warmv));
    const unsigned long long t0 = __builtin_amdgcn_s_memrealtime();
    bool bad = false;
    {
      Tile T;
      get_tile(rank, b, T);
      const bool xrem = T.xsrc != rank, wrem = T.owner != rank;
      int rdy = 0;
      if (!xrem && !wrem) { rdy = NC; if (lane == 0) lds_st(ctl + 1, (unsigned)NC); }
      unsigned it = 0;
      while (rdy < NC) {
        unsigned f = call;
        const int cx = rdy + (lane >> 1), cw = rdy + ((lane - 8) >> 2);
        if (lane < 8) { if (xrem && cx < NC) f = ld4_sys(myf + FX_W + T.mt * 32 + rdy * 2 + lane); }
        else if (lane < 24) { if (wrem && cw < NC) f = ld4_sys(myf + FW_W + T.wslot * 64 + rdy * 4 + (lane - 8)); }
        unsigned tm = 0;
        if ((++it & 15) == 0) tm = __hip_atomic_load(tmo, __ATOMIC_RELAXED, __HIP_MEMORY_SCOPE_SYSTEM);
        const unsigned bl = (unsigned)__ballot(f >= call);
        unsigned bx = bl & 0xffu, bw = (bl >> 8) & 0xffffu;
        bx &= bx >> 1; bx &= 0x55u;
        bw &= bw >> 1; bw &= bw >> 2; bw &= 0x1111u;
        const int nrx = __builtin_ctz((~bx & 0x55u) | 0x100u) >> 1;
        const int nrw = __builtin_ctz((~bw & 0x1111u) | 0x10000u) >> 2;
        int nr = rdy + (nrx < nrw ? nrx : nrw);
        nr = nr < NC ? nr : NC;
        if (tm == call || ((it & 15) == 0 && __builtin_amdgcn_s_memrealtime() - t0 > DEADLINE)) {
          if (lane == 0) { __hip_atomic_store(tmo, call, __ATOMIC_RELAXED, __HIP_MEMORY_SCOPE_SYSTEM); lds_st(ctl + 4, 1u); }
          bad = true; nr = NC;
        }
        if (nr > rdy) {
          if (lane == 0) lds_st(ctl + 1, (unsigned)nr);
          rdy = nr;
        } else {
          __builtin_amdgcn_s_sleep(1);
        }
      }
    }
    STAMP(2);
    if (b < 8) {
      bool done = false;
      while (!done) {
        const unsigned fy = ld4_sys(myf + FY_W + b * 64 + lane);
        const unsigned tm = __hip_atomic_load(tmo, __ATOMIC_RELAXED, __HIP_MEMORY_SCOPE_SYSTEM);
        done = __all(fy >= call);
        if (!done) {
          if (tm == call || __builtin_amdgcn_s_memrealtime() - t0 > DEADLINE) {
            if (lane == 0) __hip_atomic_store(tmo, call, __ATOMIC_RELAXED, __HIP_MEMORY_SCOPE_SYSTEM);
            bad = true; done = true;
          } else __builtin_amdgcn_s_sleep(1);
        }
      }
      if (bad && lane == 0) {
        ((unsigned short*)P.out[rank])[(size_t)(rank * MS) * NS + b] = 0x7fc0;
      }
      STAMP(3);
    }
    asm volatile("agk_code_end2:");
    return;
  }

  const int wm = wv >> 1, wn = wv & 1;
  const unsigned lbase = (unsigned)(size_t)(__attribute__((address_space(3))) char*)smem;
  const int r8 = lane >> 3;
  __builtin_amdgcn_s_barrier();
  asm volatile("" ::: "memory");
  unsigned warm2 = 0;
  if ((int)(tid * 64u) < code_len) warm2 = *(const unsigned*)(code_pc + tid * 64u);
  unsigned call = lds_ld(ctl + 0);
  while (call == 0) { __builtin_amdgcn_s_sleep(0); call = lds_ld(ctl + 0); }
  asm volatile("" ::: "memory");
  asm volatile("" :: "v"(warm2));
  const char* sbase = myws + (size_t)(call & 1) * PAR_BYTES;

  const int m16 = lane & 15, g4 = lane >> 4, q4 = (lane & 15) >> 2, p4 = lane & 3;
  const unsigned xa0 = wv * 2048 + m16 * 64 + ((g4 ^ ((m16 >> 2) & 3)) * 16);
  const unsigned fwl = ((q4 >> 1) & 1) | ((g4 & 1) << 2);
  const unsigned wa_0 = wv * 4096 + g4 * 1024 + q4 * 128 + ((p4 ^ fwl) * 16) + ((p4 & 1) * 8);

  {
    Tile T;
    get_tile(rank, b, T);
    const bool xrem = T.xsrc != rank, wrem = T.owner != rank;
    const char* xg; unsigned xstride_c, xstride_ks;
    if (xrem) {
      xg = sbase + (size_t)T.mt * XT_BYTES + wv * 2048 + lane * 16; xstride_c = XS_BYTES; xstride_ks = 1024;
    } else {
      const int r16 = lane >> 2;
      const int c4 = (lane & 3) ^ ((r16 >> 2) & 3);
      xg = P.x + (size_t)(T.xt * 32 + r16) * (KDIM * 2) + wv * 64 + c4 * 16; xstride_c = KC * 2; xstride_ks = 16 * KDIM * 2;
    }
    const char* wg0; const char* wg1; unsigned wstride_c, wstride_k2;
    if (wrem) {
      wg0 = sbase + XSTG_BYTES + (size_t)T.wslot * WT_BYTES + wv * 4096 + lane * 16; wg1 = wg0 + 1024;
      wstride_c = WS_BYTES; wstride_k2 = 2048;
    } else {
      const int f0 = ((r8 >> 1) & 1), f1 = f0 | 4;
      wg0 = P.w + (size_t)(wv * 32 + r8) * (NS * 2) + T.ct * 128 + (((lane & 7) ^ f0) * 16);
      wg1 = P.w + (size_t)(wv * 32 + 8 + r8) * (NS * 2) + T.ct * 128 + (((lane & 7) ^ f1) * 16);
      wstride_c = KC * NS * 2; wstride_k2 = 16 * NS * 2;
    }
    f32x4 acc[2][2][2];
#pragma unroll
    for (int i = 0; i < 8; ++i) acc[i >> 2][(i >> 1) & 1][i & 1] = f32x4{0.f, 0.f, 0.f, 0.f};
    constexpr int cbase = 0;
    int issued = 0;
    for (int c = 0; c < NC; ++c) {
      int rdy = (int)lds_ld(ctl + 1) - cbase;
      while (issued <= c && rdy <= issued) { __builtin_amdgcn_s_sleep(0); rdy = (int)lds_ld(ctl + 1) - cbase; }
      asm volatile("" ::: "memory");
      int lim = c + PFD;
      lim = lim < rdy ? lim : rdy;
      lim = lim < NC ? lim : NC;
      while (issued < lim) {
        const int sl = (cbase + issued) % NSLOT;
        const unsigned lw = lbase + LDS_W + sl * WS_BYTES + wv * 4096;
        const unsigned lx = lbase + LDS_X + sl * XS_BYTES + wv * 2048;
        const char* w0 = wg0 + (size_t)issued * wstride_c;
        const char* w1 = wg1 + (size_t)issued * wstride_c;
        const char* x0 = xg + (size_t)issued * xstride_c;
        {
        dma16<LD_AUX>(w0, lw);
        dma16<LD_AUX>(w1, lw + 1024);
        dma16<LD_AUX>(w0 + wstride_k2, lw + 2048);
        dma16<LD_AUX>(w1 + wstride_k2, lw + 3072);
        dma16<LD_AUX>(x0, lx);
        dma16<LD_AUX>(x0 + xstride_ks, lx + 1024);
        }
        ++issued;
      }
      switch (issued - c - 1) {
        case 0: WAITV(0); break;
        case 1: WAITV(6); break;
        case 2: WAITV(12); break;
        case 3: WAITV(18); break;
        default: WAITV(24); break;
      }
      {
        const int sl = c % NSLOT;
        const unsigned ax = lbase + LDS_X + sl * XS_BYTES + xa0;
        const unsigned aw = lbase + LDS_W + sl * WS_BYTES + wa_0;
        const unsigned aw00 = aw, aw01 = aw ^ 8, aw10 = aw ^ 64, aw11 = aw ^ 72;
        u32x4 xf[2]; u32x2 wf[2][2][2];
        asm volatile(
          "ds_read_b128 %0, %10\n\t"
          "ds_read_b64_tr_b16 %2, %11\n\t"
          "ds_read_b64_tr_b16 %3, %11 offset:512\n\t"
          "ds_read_b64_tr_b16 %4, %12\n\t"
          "ds_read_b64_tr_b16 %5, %12 offset:512\n\t"
          "ds_read_b128 %1, %10 offset:1024\n\t"
          "ds_read_b64_tr_b16 %6, %13\n\t"
          "ds_read_b64_tr_b16 %7, %13 offset:512\n\t"
          "ds_read_b64_tr_b16 %8, %14\n\t"
          "ds_read_b64_tr_b16 %9, %14 offset:512\n\t"
          "s_waitcnt lgkmcnt(0)"
          : "=&v"(xf[0]), "=&v"(xf[1]), "=&v"(wf[0][0][0]), "=&v"(wf[0][0][1]), "=&v"(wf[0][1][0]), "=&v"(wf[0][1][1]),
            "=&v"(wf[1][0][0]), "=&v"(wf[1][0][1]), "=&v"(wf[1][1][0]), "=&v"(wf[1][1][1])
          : "v"(ax), "v"(aw00), "v"(aw01), "v"(aw10), "v"(aw11) : "memory");
#pragma unroll
        for (int i = 0; i < 4; ++i) {
          const int n_ = i >> 1, t_ = i & 1;
          const u32x4 a = {wf[n_][t_][0][0], wf[n_][t_][0][1], wf[n_][t_][1][0], wf[n_][t_][1][1]};
          acc[0][n_][t_] = __builtin_amdgcn_mfma_f32_16x16x32_bf16(__builtin_bit_cast(bf16x8, a), __builtin_bit_cast(bf16x8, xf[0]), acc[0][n_][t_], 0, 0, 0);
          acc[1][n_][t_] = __builtin_amdgcn_mfma_f32_16x16x32_bf16(__builtin_bit_cast(bf16x8, a), __builtin_bit_cast(bf16x8, xf[1]), acc[1][n_][t_], 0, 0, 0);
        }
      }
    }
    f32x4 acc0, acc1;
    {
      f32x4* mine = (f32x4*)(smem + LDS_W + wv * 4096) + lane;
#pragma unroll
      for (int i = 0; i < 8; ++i) mine[(i >> 2) * (WS_BYTES / 16) + (i & 3) * 64] = acc[i >> 2][(i >> 1) & 1][i & 1];
      asm volatile("s_waitcnt lgkmcnt(0)" ::: "memory");
      if (lane == 0) __hip_atomic_fetch_add(ctl + 3, 1u, __ATOMIC_RELAXED, __HIP_MEMORY_SCOPE_WORKGROUP);
      while (lds_ld(ctl + 3) < 4u) { __builtin_amdgcn_s_sleep(0); }
      asm volatile("" ::: "memory");
      const int cg = lane & 7, srcl = (wv & 1) * 8 + (lane >> 3) + 16 * (cg & 3);
      const f32x4* src = (const f32x4*)(smem + LDS_W + wm * WS_BYTES + (cg >> 2) * 2048) + srcl;
      acc0 = src[0]; acc1 = src[64];
#pragma unroll
      for (int q = 1; q < 4; ++q) { acc0 += src[q * 256]; acc1 += src[q * 256 + 64]; }
    }
    const bool bad = lds_ld(ctl + 4) != 0;
    const f32x4 lo = (lane & 1) ? acc1 : acc0, hi = (lane & 1) ? acc0 : acc1;
    u32x4 o;
    o[0] = pack_relu(lo[0], lo[1]); o[1] = pack_relu(lo[2], lo[3]);
    o[2] = pack_relu(hi[0], hi[1]); o[3] = pack_relu(hi[2], hi[3]);
    if (bad) { o[0] = o[1] = o[2] = o[3] = 0x7fc07fc0u; }
    const int grow = T.xsrc * MS + T.xt * 32 + wv * 8 + (lane >> 3);
    char* op = P.out[T.owner] + (size_t)grow * (NS * 2) + T.ct * 128 + (lane & 7) * 16;
    if (!wrem) {
      st16_sys(op, o);
    } else {
      st16_sys(op, o);
      WAITV(0);
      if (lane == 0) st4_sys((unsigned*)(P.ws[T.owner] + FLAG_OFF) + FY_W + ((rank >> 1) * 32 + T.mt) * 4 + wv, call);
    }
    if (wv == 0) STAMP(4);
    asm volatile("agk_code_end0:");
  }
}

}

extern "C" void kernel_launch(int rank, int nranks, ncclComm_t comm,
                              hipStream_t stream,
                              void* const* d_in, const int* in_sizes, int n_in,
                              void* d_out, int out_size,
                              const int* in_dtypes, int out_dtype,
                              void* const* d_out_all,
                              void* const* d_ws_all, size_t ws_bytes) {
  if (nranks != NR || ws_bytes < WS_NEED || n_in < 2 || (unsigned)rank >= (unsigned)NR) {
    fprintf(stderr, "unsupported configuration\n");
    return;
  }
  Params p;
  p.x = (const char*)d_in[0];
  p.w = (const char*)d_in[1];
  for (int i = 0; i < NR; ++i) { p.out[i] = (char*)d_out_all[i]; p.ws[i] = (char*)d_ws_all[i]; }
  p.rank = rank;
  ag_gemm_kernel<<<NWG, NTHREADS, 0, stream>>>(p);
  HIP_CHECK(hipGetLastError());
}
```

```cpp
#include <hip/hip_runtime.h>
#include <rccl/rccl.h>
#include <cstdio>
#include <type_traits>

#define HIP_CHECK(expr) do { hipError_t _e = (expr); if (_e != hipSuccess) { \
    fprintf(stderr, "HIP error %s at %s:%d\n", hipGetErrorString(_e), __FILE__, __LINE__); \
    return; } } while (0)

#define MODE_AG 0
#define ACQ_MODE 2
#define PFD 5
#define NPW 1
#define NP2 64
#define NPUSHWG 256
#define WT_OUT 1
#define PHASES 0
#define XMAP 1
#define WARM 1
#define AFF 1
#define RAWP 1
#define SUBW 2
#define TAILX 0
#define CSLEEP 0
#define HALFU 0
#define WARMC 1
#define LD_AUX 0

namespace {

constexpr int NR = 8;
constexpr int KDIM = 2048;
constexpr int MS = 256, NS = 256;
constexpr int BM = 32, BN = 64, KC = 128, NC = KDIM / KC;
constexpr int NWG = 256;
constexpr int NWAVES = 5 + NPW;
constexpr int NTHREADS = NWAVES * 64;

constexpr int NMT = 32;
constexpr int PPC = 70 * (HALFU + 1);
constexpr int NPAIRS = PPC * NC;
constexpr int NP = NPUSHWG + (NPW - 1) * NP2;
static_assert(NPAIRS >= 2 * NP, "two static units per pusher wave");

constexpr size_t XT_BYTES = (size_t)BM * KDIM * 2;
constexpr size_t WT_BYTES = (size_t)BN * KDIM * 2;
constexpr size_t XSTG_BYTES = (size_t)NMT * XT_BYTES;
constexpr size_t WSTG_BYTES = MODE_AG ? 0 : 4 * WT_BYTES;
constexpr size_t PAR_BYTES = XSTG_BYTES + WSTG_BYTES;
constexpr size_t FLAG_OFF = 16u << 20;
static_assert(2 * PAR_BYTES <= FLAG_OFF, "staging overflows");
constexpr size_t WS_NEED = FLAG_OFF + (256u << 10);
constexpr int FX_W = 0;
constexpr int FW_W = 2048;
constexpr int FY_W = 4096;
constexpr int CNT_W = 8192;
constexpr int TMO_W = 8192 + 1024;
constexpr int TICKET_W = 8192 + 2048;
constexpr int DBG_W = 12288;

constexpr unsigned long long DEADLINE = 50000000ull;

constexpr int XS_BYTES = BM * KC * 2;
constexpr int WS_BYTES = BN * KC * 2;
constexpr int NSLOT = 6;
constexpr int LDS_X = 0;
constexpr int LDS_W = NSLOT * XS_BYTES;
constexpr int LDS_CTL = LDS_W + NSLOT * WS_BYTES;
constexpr int LDS_BYTES = LDS_CTL + 64;

typedef __attribute__((ext_vector_type(4))) unsigned u32x4;
typedef __attribute__((ext_vector_type(2))) unsigned u32x2;
typedef __attribute__((ext_vector_type(8))) short bf16x8;
typedef __attribute__((ext_vector_type(4))) float f32x4;
typedef __attribute__((ext_vector_type(2))) float f32x2;
typedef __attribute__((ext_vector_type(2))) __bf16 bf16x2;
typedef __attribute__((address_space(3))) unsigned lds_u32;

struct Params {
  const char* x;
  const char* w;
  char* out[NR];
  char* ws[NR];
  int rank;
};

__device__ __forceinline__ void st16_sys(void* p, u32x4 v) {
  asm volatile("global_store_dwordx4 %0, %1, off sc0 sc1\n\ts_nop 1" :: "v"(p), "v"(v) : "memory");
}
__device__ __forceinline__ void st4_sys(void* p, unsigned v) {
  asm volatile("global_store_dword %0, %1, off sc0 sc1" :: "v"(p), "v"(v) : "memory");
}
__device__ __forceinline__ unsigned ld4_sys(const void* p) {
  return __hip_atomic_load((const unsigned*)p, __ATOMIC_RELAXED, __HIP_MEMORY_SCOPE_SYSTEM);
}
__device__ __forceinline__ unsigned lds_ld(unsigned* p) {
  return __hip_atomic_load(p, __ATOMIC_RELAXED, __HIP_MEMORY_SCOPE_WORKGROUP);
}
__device__ __forceinline__ void lds_st(unsigned* p, unsigned v) {
  __hip_atomic_store(p, v, __ATOMIC_RELAXED, __HIP_MEMORY_SCOPE_WORKGROUP);
}
#define WAITV(n) asm volatile("s_waitcnt vmcnt(" #n ")" ::: "memory")

template <int AUX>
__device__ __forceinline__ void dma16(const char* g, unsigned l) {
  if constexpr (AUX == 0)
    asm volatile("s_mov_b32 m0, %1\n\ts_nop 0\n\tglobal_load_lds_dwordx4 %0, off" :: "v"(g), "s"(l) : "memory");
  else if constexpr (AUX == 16)
    asm volatile("s_mov_b32 m0, %1\n\ts_nop 0\n\tglobal_load_lds_dwordx4 %0, off sc1" :: "v"(g), "s"(l) : "memory");
  else if constexpr (AUX == 17)
    asm volatile("s_mov_b32 m0, %1\n\ts_nop 0\n\tglobal_load_lds_dwordx4 %0, off sc0 sc1" :: "v"(g), "s"(l) : "memory");
  else
    asm volatile("s_mov_b32 m0, %1\n\ts_nop 0\n\tglobal_load_lds_dwordx4 %0, off nt" :: "v"(g), "s"(l) : "memory");
}

__device__ __forceinline__ unsigned pack_relu(float a, float b) {
  a = a < 0.f ? 0.f : a;
  b = b < 0.f ? 0.f : b;
  f32x2 f = {a, b};
  bf16x2 h = __builtin_convertvector(f, bf16x2);
  return __builtin_bit_cast(unsigned, h);
}

#define STAMP(idx) do {} while (0)

struct Tile { int mt, xsrc, xt, owner, ct, wslot; };

__device__ __forceinline__ void get_tile(int rank, int b, Tile& T) {
  const int xcd = b & 7, s = b >> 3;
  const int nt = (xcd >> 2) * 4 + (s >> 3);
  const int mt = (xcd & 3) * 8 + (s & 7);
  const int par_me = rank & 1;
  T.wslot = nt - 4;
  T.owner = nt < 4 ? rank : (1 - par_me) + 2 * (nt - 4);
  T.ct = nt < 4 ? nt : (rank >> 1);
  T.mt = mt;
  if (mt < 20) { T.xsrc = par_me + 2 * (mt / 5); T.xt = mt % 5; }
  else { T.xsrc = (1 - par_me) + 2 * ((mt - 20) / 3); T.xt = 5 + (mt - 20) % 3; }
}

__global__ void __launch_bounds__(NTHREADS, 1) ag_gemm_kernel(Params P) {
  __shared__ __attribute__((aligned(1024))) char smem[LDS_BYTES];
  unsigned* ctl = (unsigned*)(smem + LDS_CTL);
  const int tid = threadIdx.x, lane = tid & 63;
  const int wv = __builtin_amdgcn_readfirstlane(tid >> 6);
  const int b = blockIdx.x;
  const int rank = P.rank;
  char* myws = P.ws[rank];
  unsigned* myf = (unsigned*)(myws + FLAG_OFF);
  unsigned* tmo = myf + TMO_W;

  if (wv == 0) STAMP(0);
  const char* code_pc; int code_len;
  asm volatile("agk_code_begin:\n\ts_getpc_b64 %0\n\ts_mov_b32 %1, agk_code_end0-agk_code_begin-8\n\t"
               "s_max_i32 %1, %1, agk_code_end1-agk_code_begin-8\n\ts_max_i32 %1, %1, agk_code_end2-agk_code_begin-8"
               : "=s"(code_pc), "=s"(code_len) :: "scc");

  if (wv >= 5) {
    const int g = wv == 5 ? b : NPUSHWG + b;
    __builtin_amdgcn_s_barrier();
    if (b >= (wv == 5 ? NPUSHWG : NP2)) return;
    unsigned call;
    asm volatile("global_load_dword %0, %1, off sc0 sc1" : "=v"(call) : "v"(myf + CNT_W + NWG + g) : "memory");
    const int r = lane >> 3;
    const int r16 = lane >> 2;
    const unsigned xl = r16 * (KDIM * 2) + ((lane & 3) ^ ((r16 >> 2) & 3)) * 16;
    const unsigned wl0 = r * (NS * 2) + ((lane & 7) ^ ((r >> 1) & 1)) * 16;
    const unsigned wl1 = (r + 8) * (NS * 2) + ((lane & 7) ^ (((r >> 1) & 1) | 4)) * 16;
    const int dsel = g % 7;
    struct Unit { u32x4 v[4]; char* dp; unsigned* fp; int idx; };
    auto fetch = [&](int i, Unit& U) {
      const int c = i / (PPC / 7);
      const int d = 1 + dsel, u = i % (PPC / 7);
      const int dest = (rank + d) & 7;
      char* dws = P.ws[dest];
      bool isw = false; int xu = u;
      if (d & 1) { if (u < 4) isw = true; else xu = 10 + (u - 4); }
      const int t = xu >> 1, h = xu & 1;
      const int dmt = t < 5 ? (rank >> 1) * 5 + t : 20 + (rank >> 1) * 3 + (t - 5);
      const char* sb = isw ? P.w + (size_t)(c * KC + u * 32) * (NS * 2) + (dest >> 1) * 128
                           : P.x + (size_t)(t * 32) * (KDIM * 2) + c * (KC * 2) + h * 128;
      const unsigned o0 = isw ? wl0 : xl, o1 = isw ? wl1 : xl + 16 * KDIM * 2;
      const unsigned o2 = isw ? wl0 + 16 * NS * 2 : xl + 64, o3 = isw ? wl1 + 16 * NS * 2 : xl + 64 + 16 * KDIM * 2;
      asm volatile("global_load_dwordx4 %0, %1, off" : "=v"(U.v[0]) : "v"(sb + o0) : "memory");
      asm volatile("global_load_dwordx4 %0, %1, off" : "=v"(U.v[1]) : "v"(sb + o1) : "memory");
      asm volatile("global_load_dwordx4 %0, %1, off" : "=v"(U.v[2]) : "v"(sb + o2) : "memory");
      asm volatile("global_load_dwordx4 %0, %1, off" : "=v"(U.v[3]) : "v"(sb + o3) : "memory");
      U.idx = i;
      U.dp = dws + (isw ? (unsigned)(XSTG_BYTES + (rank >> 1) * WT_BYTES) + c * WS_BYTES + u * 4096
                        : (unsigned)(dmt * XT_BYTES) + c * XS_BYTES + h * 4096) + lane * 16;
      U.fp = (unsigned*)(dws + FLAG_OFF) + (isw ? FW_W + ((rank >> 1) * NC + c) * 4 + u : FX_W + (dmt * NC + c) * 2 + h);
    };
    Unit UA, UB;
    const int g0 = g / 7, NPL = NP / 7 + (g % 7 < NP % 7 ? 1 : 0), NPAIRSL = NPAIRS / 7;
    unsigned* ticket = myf + TICKET_W + dsel * 64;
    fetch(g0, UA);
    fetch(g0 + NPL, UB);
    WAITV(4);
    asm volatile("" : "+v"(call));
    call = __builtin_amdgcn_readfirstlane(call) + 1;
    if (lane == 0) st4_sys(myf + CNT_W + NWG + g, call);
    const size_t poff = (size_t)(call & 1) * PAR_BYTES;
    const unsigned tbase = (call - 1) * (unsigned)(NPAIRSL - NPL) - 2u * NPL;
    STAMP(6);
    int nxt = g0 + NPL;
    auto step = [&](Unit& U) -> bool {
      char* dp = U.dp + poff;
#pragma unroll
      for (int ks = 0; ks < (HALFU ? 2 : 4); ++ks) {
        st16_sys(dp + ks * 1024, U.v[ks]);
        if ((ks == 1 || ks == 2) && (TAILX == 0 || U.idx < NPAIRSL - TAILX)) WAITV(1);
      }
      unsigned tk = 0;
      const bool more = nxt < NPAIRSL;
      if (more && lane == 0) tk = __hip_atomic_fetch_add(ticket, 1u, __ATOMIC_RELAXED, __HIP_MEMORY_SCOPE_SYSTEM);
      WAITV(0);
      if (lane == 0) st4_sys(U.fp, call);
      if (!more) return false;
      nxt = (int)(__builtin_amdgcn_readfirstlane(tk) - tbase);
      if (nxt < NPAIRSL) fetch(nxt, U);
      return true;
    };
    while (true) {
      if (!step(UA)) break;
      if (!step(UB)) break;
    }
    STAMP(1);
    asm volatile("agk_code_end1:");
    return;
  }

  if (wv == 4) {
    unsigned call = 0;
    if (lane < 8) lds_st(ctl + lane, 0u);
    asm volatile("s_waitcnt lgkmcnt(0)" ::: "memory");
    __builtin_amdgcn_s_barrier();
    unsigned warmv = 0;
    {
      Tile T;
      get_tile(rank, b, T);
      if (lane == 1) warmv = ld4_sys(P.out[T.owner] + (size_t)(T.xsrc * MS + T.xt * 32) * (NS * 2) + T.ct * 128);
      if (lane == 2) warmv = ld4_sys((unsigned*)(P.ws[T.owner] + FLAG_OFF) + FY_W);
    }
    if (lane == 0) {
      call = ld4_sys(myf + CNT_W + b) + 1;
      st4_sys(myf + CNT_W + b, call);
    }
    call = __builtin_amdgcn_readfirstlane(call);
    __builtin_amdgcn_fence(__ATOMIC_ACQUIRE, "");
    asm volatile("s_waitcnt vmcnt(0)" ::: "memory");
    if (lane == 0) lds_st(ctl + 0, call);
    asm volatile("" :: "v"(warmv));
    const unsigned long long t0 = __builtin_amdgcn_s_memrealtime();
    bool bad = false;
    {
      Tile T;
      get_tile(rank, b, T);
      const bool xrem = T.xsrc != rank, wrem = T.owner != rank;
      int rdy = 0;
      if (!xrem && !wrem) { rdy = NC; if (lane == 0) lds_st(ctl + 1, (unsigned)NC); }
      unsigned it = 0;
      while (rdy < NC) {
        unsigned f = call;
        const int cx = rdy + (lane >> 1), cw = rdy + ((lane - 8) >> 2);
        if (lane < 8) { if (xrem && cx < NC) f = ld4_sys(myf + FX_W + T.mt * 32 + rdy * 2 + lane); }
        else if (lane < 24) { if (wrem && cw < NC) f = ld4_sys(myf + FW_W + T.wslot * 64 + rdy * 4 + (lane - 8)); }
        unsigned tm = 0;
        if ((++it & 15) == 0) tm = __hip_atomic_load(tmo, __ATOMIC_RELAXED, __HIP_MEMORY_SCOPE_SYSTEM);
        const unsigned bl = (unsigned)__ballot(f >= call);
        unsigned bx = bl & 0xffu, bw = (bl >> 8) & 0xffffu;
        bx &= bx >> 1; bx &= 0x55u;
        bw &= bw >> 1; bw &= bw >> 2; bw &= 0x1111u;
        const int nrx = __builtin_ctz((~bx & 0x55u) | 0x100u) >> 1;
        const int nrw = __builtin_ctz((~bw & 0x1111u) | 0x10000u) >> 2;
        int nr = rdy + (nrx < nrw ? nrx : nrw);
        nr = nr < NC ? nr : NC;
        if (tm == call || ((it & 15) == 0 && __builtin_amdgcn_s_memrealtime() - t0 > DEADLINE)) {
          if (lane == 0) { __hip_atomic_store(tmo, call, __ATOMIC_RELAXED, __HIP_MEMORY_SCOPE_SYSTEM); lds_st(ctl + 4, 1u); }
          bad = true; nr = NC;
        }
        if (nr > rdy) {
          if (lane == 0) lds_st(ctl + 1, (unsigned)nr);
          rdy = nr;
        } else {
          __builtin_amdgcn_s_sleep(1);
        }
      }
    }
    STAMP(2);
    if (b < 8) {
      bool done = false;
      while (!done) {
        const unsigned fy = ld4_sys(myf + FY_W + b * 64 + lane);
        const unsigned tm = __hip_atomic_load(tmo, __ATOMIC_RELAXED, __HIP_MEMORY_SCOPE_SYSTEM);
        done = __all(fy >= call);
        if (!done) {
          if (tm == call || __builtin_amdgcn_s_memrealtime() - t0 > DEADLINE) {
            if (lane == 0) __hip_atomic_store(tmo, call, __ATOMIC_RELAXED, __HIP_MEMORY_SCOPE_SYSTEM);
            bad = true; done = true;
          } else __builtin_amdgcn_s_sleep(1);
        }
      }
      if (bad && lane == 0) {
        ((unsigned short*)P.out[rank])[(size_t)(rank * MS) * NS + b] = 0x7fc0;
      }
      STAMP(3);
    }
    asm volatile("agk_code_end2:");
    return;
  }

  const int wm = wv >> 1, wn = wv & 1;
  const unsigned lbase = (unsigned)(size_t)(__attribute__((address_space(3))) char*)smem;
  const int r8 = lane >> 3;
  __builtin_amdgcn_s_barrier();
  asm volatile("" ::: "memory");
  unsigned warm2 = 0;
  if ((int)(tid * 64u) < code_len) warm2 = *(const unsigned*)(code_pc + tid * 64u);
  unsigned call = lds_ld(ctl + 0);
  while (call == 0) { __builtin_amdgcn_s_sleep(0); call = lds_ld(ctl + 0); }
  asm volatile("" ::: "memory");
  asm volatile("" :: "v"(warm2));
  const char* sbase = myws + (size_t)(call & 1) * PAR_BYTES;

  const int m16 = lane & 15, g4 = lane >> 4, q4 = (lane & 15) >> 2, p4 = lane & 3;
  const unsigned xa0 = wv * 2048 + m16 * 64 + ((g4 ^ ((m16 >> 2) & 3)) * 16);
  const unsigned fwl = ((q4 >> 1) & 1) | ((g4 & 1) << 2);
  const unsigned wa_0 = wv * 4096 + g4 * 1024 + q4 * 128 + ((p4 ^ fwl) * 16) + ((p4 & 1) * 8);

  {
    Tile T;
    get_tile(rank, b, T);
    const bool xrem = T.xsrc != rank, wrem = T.owner != rank;
    const char* xg; unsigned xstride_c, xstride_ks;
    if (xrem) {
      xg = sbase + (size_t)T.mt * XT_BYTES + wv * 2048 + lane * 16; xstride_c = XS_BYTES; xstride_ks = 1024;
    } else {
      const int r16 = lane >> 2;
      const int c4 = (lane & 3) ^ ((r16 >> 2) & 3);
      xg = P.x + (size_t)(T.xt * 32 + r16) * (KDIM * 2) + wv * 64 + c4 * 16; xstride_c = KC * 2; xstride_ks = 16 * KDIM * 2;
    }
    const char* wg0; const char* wg1; unsigned wstride_c, wstride_k2;
    if (wrem) {
      wg0 = sbase + XSTG_BYTES + (size_t)T.wslot * WT_BYTES + wv * 4096 + lane * 16; wg1 = wg0 + 1024;
      wstride_c = WS_BYTES; wstride_k2 = 2048;
    } else {
      const int f0 = ((r8 >> 1) & 1), f1 = f0 | 4;
      wg0 = P.w + (size_t)(wv * 32 + r8) * (NS * 2) + T.ct * 128 + (((lane & 7) ^ f0) * 16);
      wg1 = P.w + (size_t)(wv * 32 + 8 + r8) * (NS * 2) + T.ct * 128 + (((lane & 7) ^ f1) * 16);
      wstride_c = KC * NS * 2; wstride_k2 = 16 * NS * 2;
    }
    f32x4 acc[2][2][2];
#pragma unroll
    for (int i = 0; i < 8; ++i) acc[i >> 2][(i >> 1) & 1][i & 1] = f32x4{0.f, 0.f, 0.f, 0.f};
    constexpr int cbase = 0;
    int issued = 0;
    for (int c = 0; c < NC; ++c) {
      int rdy = (int)lds_ld(ctl + 1) - cbase;
      while (issued <= c && rdy <= issued) {
        __builtin_amdgcn_s_sleep(CSLEEP); rdy = (int)lds_ld(ctl + 1) - cbase;
      }
      asm volatile("" ::: "memory");
      int lim = c + PFD;
      lim = lim < rdy ? lim : rdy;
      lim = lim < NC ? lim : NC;
      while (issued < lim) {
        const int sl = (cbase + issued) % NSLOT;
        const unsigned lw = lbase + LDS_W + sl * WS_BYTES + wv * 4096;
        const unsigned lx = lbase + LDS_X + sl * XS_BYTES + wv * 2048;
        const char* w0 = wg0 + (size_t)issued * wstride_c;
        const char* w1 = wg1 + (size_t)issued * wstride_c;
        const char* x0 = xg + (size_t)issued * xstride_c;
        {
        dma16<LD_AUX>(w0, lw);
        dma16<LD_AUX>(w1, lw + 1024);
        dma16<LD_AUX>(w0 + wstride_k2, lw + 2048);
        dma16<LD_AUX>(w1 + wstride_k2, lw + 3072);
        dma16<LD_AUX>(x0, lx);
        dma16<LD_AUX>(x0 + xstride_ks, lx + 1024);
        }
        ++issued;
      }
      switch (issued - c - 1) {
        case 0: WAITV(0); break;
        case 1: WAITV(6); break;
        case 2: WAITV(12); break;
        case 3: WAITV(18); break;
        default: WAITV(24); break;
      }
      {
        const int sl = c % NSLOT;
        const unsigned ax = lbase + LDS_X + sl * XS_BYTES + xa0;
        const unsigned aw = lbase + LDS_W + sl * WS_BYTES + wa_0;
        const unsigned aw00 = aw, aw01 = aw ^ 8, aw10 = aw ^ 64, aw11 = aw ^ 72;
        u32x4 xf[2]; u32x2 wf[2][2][2];
        asm volatile(
          "ds_read_b128 %0, %10\n\t"
          "ds_read_b64_tr_b16 %2, %11\n\t"
          "ds_read_b64_tr_b16 %3, %11 offset:512\n\t"
          "ds_read_b64_tr_b16 %4, %12\n\t"
          "ds_read_b64_tr_b16 %5, %12 offset:512\n\t"
          "ds_read_b128 %1, %10 offset:1024\n\t"
          "ds_read_b64_tr_b16 %6, %13\n\t"
          "ds_read_b64_tr_b16 %7, %13 offset:512\n\t"
          "ds_read_b64_tr_b16 %8, %14\n\t"
          "ds_read_b64_tr_b16 %9, %14 offset:512\n\t"
          "s_waitcnt lgkmcnt(0)"
          : "=&v"(xf[0]), "=&v"(xf[1]), "=&v"(wf[0][0][0]), "=&v"(wf[0][0][1]), "=&v"(wf[0][1][0]), "=&v"(wf[0][1][1]),
            "=&v"(wf[1][0][0]), "=&v"(wf[1][0][1]), "=&v"(wf[1][1][0]), "=&v"(wf[1][1][1])
          : "v"(ax), "v"(aw00), "v"(aw01), "v"(aw10), "v"(aw11) : "memory");
#pragma unroll
        for (int i = 0; i < 4; ++i) {
          const int n_ = i >> 1, t_ = i & 1;
          const u32x4 a = {wf[n_][t_][0][0], wf[n_][t_][0][1], wf[n_][t_][1][0], wf[n_][t_][1][1]};
          acc[0][n_][t_] = __builtin_amdgcn_mfma_f32_16x16x32_bf16(__builtin_bit_cast(bf16x8, a), __builtin_bit_cast(bf16x8, xf[0]), acc[0][n_][t_], 0, 0, 0);
          acc[1][n_][t_] = __builtin_amdgcn_mfma_f32_16x16x32_bf16(__builtin_bit_cast(bf16x8, a), __builtin_bit_cast(bf16x8, xf[1]), acc[1][n_][t_], 0, 0, 0);
        }
      }
    }
    f32x4 acc0, acc1;
    {
      f32x4* mine = (f32x4*)(smem + LDS_W + wv * 4096) + lane;
#pragma unroll
      for (int i = 0; i < 8; ++i) mine[(i >> 2) * (WS_BYTES / 16) + (i & 3) * 64] = acc[i >> 2][(i >> 1) & 1][i & 1];
      asm volatile("s_waitcnt lgkmcnt(0)" ::: "memory");
      if (lane == 0) __hip_atomic_fetch_add(ctl + 3, 1u, __ATOMIC_RELAXED, __HIP_MEMORY_SCOPE_WORKGROUP);
      while (lds_ld(ctl + 3) < 4u) { __builtin_amdgcn_s_sleep(0); }
      asm volatile("" ::: "memory");
      const int cg = lane & 7, srcl = (wv & 1) * 8 + (lane >> 3) + 16 * (cg & 3);
      const f32x4* src = (const f32x4*)(smem + LDS_W + wm * WS_BYTES + (cg >> 2) * 2048) + srcl;
      acc0 = src[0]; acc1 = src[64];
#pragma unroll
      for (int q = 1; q < 4; ++q) { acc0 += src[q * 256]; acc1 += src[q * 256 + 64]; }
    }
    const bool bad = lds_ld(ctl + 4) != 0;
    const f32x4 lo = (lane & 1) ? acc1 : acc0, hi = (lane & 1) ? acc0 : acc1;
    u32x4 o;
    o[0] = pack_relu(lo[0], lo[1]); o[1] = pack_relu(lo[2], lo[3]);
    o[2] = pack_relu(hi[0], hi[1]); o[3] = pack_relu(hi[2], hi[3]);
    if (bad) { o[0] = o[1] = o[2] = o[3] = 0x7fc07fc0u; }
    const int grow = T.xsrc * MS + T.xt * 32 + wv * 8 + (lane >> 3);
    char* op = P.out[T.owner] + (size_t)grow * (NS * 2) + T.ct * 128 + (lane & 7) * 16;
    if (!wrem) {
      st16_sys(op, o);
    } else {
      st16_sys(op, o);
      WAITV(0);
      if (lane == 0) st4_sys((unsigned*)(P.ws[T.owner] + FLAG_OFF) + FY_W + ((rank >> 1) * 32 + T.mt) * 4 + wv, call);
    }
    if (wv == 0) STAMP(4);
    asm volatile("agk_code_end0:");
  }
}

}

extern "C" void kernel_launch(int rank, int nranks, ncclComm_t comm,
                              hipStream_t stream,
                              void* const* d_in, const int* in_sizes, int n_in,
                              void* d_out, int out_size,
                              const int* in_dtypes, int out_dtype,
                              void* const* d_out_all,
                              void* const* d_ws_all, size_t ws_bytes) {
  if (nranks != NR || ws_bytes < WS_NEED || n_in < 2 || (unsigned)rank >= (unsigned)NR) {
    fprintf(stderr, "unsupported configuration\n");
    return;
  }
  Params p;
  p.x = (const char*)d_in[0];
  p.w = (const char*)d_in[1];
  for (int i = 0; i < NR; ++i) { p.out[i] = (char*)d_out_all[i]; p.ws[i] = (char*)d_ws_all[i]; }
  p.rank = rank;
  ag_gemm_kernel<<<NWG, NTHREADS, 0, stream>>>(p);
  HIP_CHECK(hipGetLastError());
}
```

```cpp
#include <hip/hip_runtime.h>
#include <rccl/rccl.h>
#include <cstdio>
#include <type_traits>

#define HIP_CHECK(expr) do { hipError_t _e = (expr); if (_e != hipSuccess) { \
    fprintf(stderr, "HIP error %s at %s:%d\n", hipGetErrorString(_e), __FILE__, __LINE__); \
    return; } } while (0)

#define MODE_AG 0
#define ACQ_MODE 2
#define PFD 2
#define NPW 1
#define NP2 64
#define NPUSHWG 256
#define WT_OUT 1
#define PHASES 0
#define XMAP 5
#define WARM 1
#define AFF 1
#define RAWP 1
#define SUBW 2
#define TAILX 0
#define CSLEEP 0
#define HALFU 0
#define WARMC 1
#define LD_AUX 0

namespace {

constexpr int NR = 8;
constexpr int KDIM = 2048;
constexpr int MS = 256, NS = 256;
constexpr int BM = 32, BN = 64, KC = 128, NC = KDIM / KC;
constexpr int NWG = 256;
constexpr int NWAVES = 5 + NPW;
constexpr int NTHREADS = NWAVES * 64;

constexpr int NMT = 32;
constexpr int PPC = 70 * (HALFU + 1);
constexpr int NPAIRS = PPC * NC;
constexpr int NP = NPUSHWG + (NPW - 1) * NP2;
static_assert(NPAIRS >= 2 * NP, "two static units per pusher wave");

constexpr size_t XT_BYTES = (size_t)BM * KDIM * 2;
constexpr size_t WT_BYTES = (size_t)BN * KDIM * 2;
constexpr size_t XSTG_BYTES = (size_t)NMT * XT_BYTES;
constexpr size_t WSTG_BYTES = MODE_AG ? 0 : 4 * WT_BYTES;
constexpr size_t PAR_BYTES = XSTG_BYTES + WSTG_BYTES;
constexpr size_t FLAG_OFF = 16u << 20;
static_assert(2 * PAR_BYTES <= FLAG_OFF, "staging overflows");
constexpr size_t WS_NEED = FLAG_OFF + (256u << 10);
constexpr int FX_W = 0;
constexpr int FW_W = 2048;
constexpr int FY_W = 4096;
constexpr int CNT_W = 8192;
constexpr int TMO_W = 8192 + 1024;
constexpr int TICKET_W = 8192 + 2048;
constexpr int DBG_W = 12288;

constexpr unsigned long long DEADLINE = 50000000ull;

constexpr int XS_BYTES = BM * KC * 2;
constexpr int WS_BYTES = BN * KC * 2;
constexpr int NSLOT = 6;
constexpr int LDS_X = 0;
constexpr int LDS_W = NSLOT * XS_BYTES;
constexpr int LDS_CTL = LDS_W + NSLOT * WS_BYTES;
constexpr int LDS_BYTES = LDS_CTL + 64;

typedef __attribute__((ext_vector_type(4))) unsigned u32x4;
typedef __attribute__((ext_vector_type(2))) unsigned u32x2;
typedef __attribute__((ext_vector_type(8))) short bf16x8;
typedef __attribute__((ext_vector_type(4))) float f32x4;
typedef __attribute__((ext_vector_type(2))) float f32x2;
typedef __attribute__((ext_vector_type(2))) __bf16 bf16x2;
typedef __attribute__((address_space(3))) unsigned lds_u32;

struct Params {
  const char* x;
  const char* w;
  char* myws;
  int rank;
  char* wsr[NR];
  char* out[NR];
  char* ws[NR];
};

__device__ __forceinline__ void st16_sys(void* p, u32x4 v) {
  asm volatile("global_store_dwordx4 %0, %1, off sc0 sc1\n\ts_nop 1" :: "v"(p), "v"(v) : "memory");
}
__device__ __forceinline__ void st4_sys(void* p, unsigned v) {
  asm volatile("global_store_dword %0, %1, off sc0 sc1" :: "v"(p), "v"(v) : "memory");
}
__device__ __forceinline__ unsigned ld4_sys(const void* p) {
  return __hip_atomic_load((const unsigned*)p, __ATOMIC_RELAXED, __HIP_MEMORY_SCOPE_SYSTEM);
}
__device__ __forceinline__ unsigned lds_ld(unsigned* p) {
  return __hip_atomic_load(p, __ATOMIC_RELAXED, __HIP_MEMORY_SCOPE_WORKGROUP);
}
__device__ __forceinline__ void lds_st(unsigned* p, unsigned v) {
  __hip_atomic_store(p, v, __ATOMIC_RELAXED, __HIP_MEMORY_SCOPE_WORKGROUP);
}
#define WAITV(n) asm volatile("s_waitcnt vmcnt(" #n ")" ::: "memory")

template <int AUX>
__device__ __forceinline__ void dma16(const char* g, unsigned l) {
  if constexpr (AUX == 0)
    asm volatile("s_mov_b32 m0, %1\n\ts_nop 0\n\tglobal_load_lds_dwordx4 %0, off" :: "v"(g), "s"(l) : "memory");
  else if constexpr (AUX == 16)
    asm volatile("s_mov_b32 m0, %1\n\ts_nop 0\n\tglobal_load_lds_dwordx4 %0, off sc1" :: "v"(g), "s"(l) : "memory");
  else if constexpr (AUX == 17)
    asm volatile("s_mov_b32 m0, %1\n\ts_nop 0\n\tglobal_load_lds_dwordx4 %0, off sc0 sc1" :: "v"(g), "s"(l) : "memory");
  else
    asm volatile("s_mov_b32 m0, %1\n\ts_nop 0\n\tglobal_load_lds_dwordx4 %0, off nt" :: "v"(g), "s"(l) : "memory");
}

__device__ __forceinline__ unsigned pack_relu(float a, float b) {
  a = a < 0.f ? 0.f : a;
  b = b < 0.f ? 0.f : b;
  f32x2 f = {a, b};
  bf16x2 h = __builtin_convertvector(f, bf16x2);
  return __builtin_bit_cast(unsigned, h);
}

#define STAMP(idx) do {} while (0)

struct Tile { int mt, xsrc, xt, owner, ct, wslot; };

__device__ __forceinline__ void get_tile(int rank, int b, Tile& T) {
  const int xcd = b & 7, s = b >> 3;
  const int nt = ((s >> 3) & 1) + 2 * (xcd >> 2) + 4 * (s >> 4);
  const int mt = (xcd & 3) + 4 * (s & 7);
  const int par_me = rank & 1;
  T.wslot = nt - 4;
  T.owner = nt < 4 ? rank : (1 - par_me) + 2 * (nt - 4);
  T.ct = nt < 4 ? nt : (rank >> 1);
  T.mt = mt;
  if (mt < 20) { T.xsrc = par_me + 2 * (mt / 5); T.xt = mt % 5; }
  else { T.xsrc = (1 - par_me) + 2 * ((mt - 20) / 3); T.xt = 5 + (mt - 20) % 3; }
}

__global__ void __launch_bounds__(NTHREADS, 1) ag_gemm_kernel(Params P) {
  __shared__ __attribute__((aligned(1024))) char smem[LDS_BYTES];
  unsigned* ctl = (unsigned*)(smem + LDS_CTL);
  const int tid = threadIdx.x, lane = tid & 63;
  const int wv = __builtin_amdgcn_readfirstlane(tid >> 6);
  const int b = blockIdx.x;
  const int rank = P.rank;
  char* myws = P.myws;
  unsigned* myf = (unsigned*)(myws + FLAG_OFF);
  unsigned* tmo = myf + TMO_W;

  if (wv == 0) STAMP(0);
  const char* code_pc; int code_len;
  asm volatile("agk_code_begin:\n\ts_getpc_b64 %0\n\ts_mov_b32 %1, agk_code_end0-agk_code_begin-8\n\t"
               "s_max_i32 %1, %1, agk_code_end1-agk_code_begin-8\n\ts_max_i32 %1, %1, agk_code_end2-agk_code_begin-8"
               : "=s"(code_pc), "=s"(code_len) :: "scc");

  if (wv >= 5) {
    const int g = wv == 5 ? b : NPUSHWG + b;
    __builtin_amdgcn_s_barrier();
    if (b >= (wv == 5 ? NPUSHWG : NP2)) return;
    unsigned call;
    asm volatile("global_load_dword %0, %1, off sc0 sc1" : "=v"(call) : "v"(myf + CNT_W + NWG + g) : "memory");
    const int r = lane >> 3;
    const int r16 = lane >> 2;
    const unsigned xl = r16 * (KDIM * 2) + ((lane & 3) ^ ((r16 >> 2) & 3)) * 16;
    const unsigned wl0 = r * (NS * 2) + ((lane & 7) ^ ((r >> 1) & 1)) * 16;
    const unsigned wl1 = (r + 8) * (NS * 2) + ((lane & 7) ^ (((r >> 1) & 1) | 4)) * 16;
    const int dsel = g % 7;
    struct Unit { u32x4 v[4]; char* dp; unsigned* fp; int idx; };
    auto fetch = [&](int i, Unit& U) {
      const int c = i / (PPC / 7);
      const int d = 1 + dsel, u = i % (PPC / 7);
      const int dest = (rank + d) & 7;
      char* dws = P.wsr[d];
      bool isw = false; int xu = u;
      if (d & 1) { if (u < 4) isw = true; else xu = 10 + (u - 4); }
      const int t = xu >> 1, h = xu & 1;
      const int dmt = t < 5 ? (rank >> 1) * 5 + t : 20 + (rank >> 1) * 3 + (t - 5);
      const char* sb = isw ? P.w + (size_t)(c * KC + u * 32) * (NS * 2) + (dest >> 1) * 128
                           : P.x + (size_t)(t * 32) * (KDIM * 2) + c * (KC * 2) + h * 128;
      const unsigned o0 = isw ? wl0 : xl, o1 = isw ? wl1 : xl + 16 * KDIM * 2;
      const unsigned o2 = isw ? wl0 + 16 * NS * 2 : xl + 64, o3 = isw ? wl1 + 16 * NS * 2 : xl + 64 + 16 * KDIM * 2;
      asm volatile("global_load_dwordx4 %0, %1, off" : "=v"(U.v[0]) : "v"(sb + o0) : "memory");
      asm volatile("global_load_dwordx4 %0, %1, off" : "=v"(U.v[1]) : "v"(sb + o1) : "memory");
      asm volatile("global_load_dwordx4 %0, %1, off" : "=v"(U.v[2]) : "v"(sb + o2) : "memory");
      asm volatile("global_load_dwordx4 %0, %1, off" : "=v"(U.v[3]) : "v"(sb + o3) : "memory");
      U.idx = i;
      U.dp = dws + (isw ? (unsigned)(XSTG_BYTES + (rank >> 1) * WT_BYTES) + c * WS_BYTES + u * 4096
                        : (unsigned)(dmt * XT_BYTES) + c * XS_BYTES + h * 4096) + lane * 16;
      U.fp = (unsigned*)(dws + FLAG_OFF) + (isw ? FW_W + ((rank >> 1) * NC + c) * 4 + u : FX_W + (dmt * NC + c) * 2 + h);
    };
    Unit UA, UB;
    const int g0 = g / 7, NPL = NP / 7 + (g % 7 < NP % 7 ? 1 : 0), NPAIRSL = NPAIRS / 7;
    unsigned* ticket = myf + TICKET_W + dsel * 64;
    fetch(g0, UA);
    fetch(g0 + NPL, UB);
    WAITV(4);
    asm volatile("" : "+v"(call));
    call = __builtin_amdgcn_readfirstlane(call) + 1;
    if (lane == 0) st4_sys(myf + CNT_W + NWG + g, call);
    const size_t poff = (size_t)(call & 1) * PAR_BYTES;
    const unsigned tbase = (call - 1) * (unsigned)(NPAIRSL - NPL) - 2u * NPL;
    STAMP(6);
    int nxt = g0 + NPL;
    auto step = [&](Unit& U) -> bool {
      char* dp = U.dp + poff;
#pragma unroll
      for (int ks = 0; ks < (HALFU ? 2 : 4); ++ks) {
        st16_sys(dp + ks * 1024, U.v[ks]);
        if ((ks == 1 || ks == 2) && (TAILX == 0 || U.idx < NPAIRSL - TAILX)) WAITV(1);
      }
      unsigned tk = 0;
      const bool more = nxt < NPAIRSL;
      if (more && lane == 0) tk = __hip_atomic_fetch_add(ticket, 1u, __ATOMIC_RELAXED, __HIP_MEMORY_SCOPE_SYSTEM);
      WAITV(0);
      if (lane == 0) st4_sys(U.fp, call);
      if (!more) return false;
      nxt = (int)(__builtin_amdgcn_readfirstlane(tk) - tbase);
      if (nxt < NPAIRSL) fetch(nxt, U);
      return true;
    };
    while (true) {
      if (!step(UA)) break;
      if (!step(UB)) break;
    }
    STAMP(1);
    asm volatile("agk_code_end1:");
    return;
  }

  if (wv == 4) {
    unsigned call = 0;
    if (lane < 8) lds_st(ctl + lane, 0u);
    asm volatile("s_waitcnt lgkmcnt(0)" ::: "memory");
    __builtin_amdgcn_s_barrier();
    unsigned warmv = 0;
    {
      Tile T;
      get_tile(rank, b, T);
      if (lane == 1) warmv = ld4_sys(P.out[T.owner] + (size_t)(T.xsrc * MS + T.xt * 32) * (NS * 2) + T.ct * 128);
      if (lane == 2) warmv = ld4_sys((unsigned*)(P.ws[T.owner] + FLAG_OFF) + FY_W);
    }
    if (lane == 0) {
      call = ld4_sys(myf + CNT_W + b) + 1;
      st4_sys(myf + CNT_W + b, call);
    }
    call = __builtin_amdgcn_readfirstlane(call);
    __builtin_amdgcn_fence(__ATOMIC_ACQUIRE, "");
    asm volatile("s_waitcnt vmcnt(0)" ::: "memory");
    if (lane == 0) lds_st(ctl + 0, call);
    asm volatile("" :: "v"(warmv));
    const unsigned long long t0 = __builtin_amdgcn_s_memrealtime();
    bool bad = false;
    {
      Tile T;
      get_tile(rank, b, T);
      const bool xrem = T.xsrc != rank, wrem = T.owner != rank;
      int rdy = 0;
      if (!xrem && !wrem) { rdy = NC; if (lane == 0) lds_st(ctl + 1, (unsigned)NC); }
      unsigned it = 0;
      while (rdy < NC) {
        unsigned f = call;
        const int cx = rdy + (lane >> 1), cw = rdy + ((lane - 8) >> 2);
        if (lane < 8) { if (xrem && cx < NC) f = ld4_sys(myf + FX_W + T.mt * 32 + rdy * 2 + lane); }
        else if (lane < 24) { if (wrem && cw < NC) f = ld4_sys(myf + FW_W + T.wslot * 64 + rdy * 4 + (lane - 8)); }
        unsigned tm = 0;
        if ((++it & 15) == 0) tm = __hip_atomic_load(tmo, __ATOMIC_RELAXED, __HIP_MEMORY_SCOPE_SYSTEM);
        const unsigned bl = (unsigned)__ballot(f >= call);
        unsigned bx = bl & 0xffu, bw = (bl >> 8) & 0xffffu;
        bx &= bx >> 1; bx &= 0x55u;
        bw &= bw >> 1; bw &= bw >> 2; bw &= 0x1111u;
        const int nrx = __builtin_ctz((~bx & 0x55u) | 0x100u) >> 1;
        const int nrw = __builtin_ctz((~bw & 0x1111u) | 0x10000u) >> 2;
        int nr = rdy + (nrx < nrw ? nrx : nrw);
        nr = nr < NC ? nr : NC;
        if (tm == call || ((it & 15) == 0 && __builtin_amdgcn_s_memrealtime() - t0 > DEADLINE)) {
          if (lane == 0) { __hip_atomic_store(tmo, call, __ATOMIC_RELAXED, __HIP_MEMORY_SCOPE_SYSTEM); lds_st(ctl + 4, 1u); }
          bad = true; nr = NC;
        }
        if (nr > rdy) {
          if (lane == 0) lds_st(ctl + 1, (unsigned)nr);
          rdy = nr;
        } else {
          __builtin_amdgcn_s_sleep(1);
        }
      }
    }
    STAMP(2);
    if (b < 8) {
      bool done = false;
      while (!done) {
        const unsigned fy = ld4_sys(myf + FY_W + b * 64 + lane);
        const unsigned tm = __hip_atomic_load(tmo, __ATOMIC_RELAXED, __HIP_MEMORY_SCOPE_SYSTEM);
        done = __all(fy >= call);
        if (!done) {
          if (tm == call || __builtin_amdgcn_s_memrealtime() - t0 > DEADLINE) {
            if (lane == 0) __hip_atomic_store(tmo, call, __ATOMIC_RELAXED, __HIP_MEMORY_SCOPE_SYSTEM);
            bad = true; done = true;
          } else __builtin_amdgcn_s_sleep(1);
        }
      }
      if (bad && lane == 0) {
        ((unsigned short*)P.out[rank])[(size_t)(rank * MS) * NS + b] = 0x7fc0;
      }
      STAMP(3);
    }
    asm volatile("agk_code_end2:");
    return;
  }

  const int wm = wv >> 1, wn = wv & 1;
  const unsigned lbase = (unsigned)(size_t)(__attribute__((address_space(3))) char*)smem;
  const int r8 = lane >> 3;
  __builtin_amdgcn_s_barrier();
  asm volatile("" ::: "memory");
  unsigned warm2 = 0;
  if ((int)(tid * 64u) < code_len) warm2 = *(const unsigned*)(code_pc + tid * 64u);
  unsigned call = lds_ld(ctl + 0);
  while (call == 0) { __builtin_amdgcn_s_sleep(0); call = lds_ld(ctl + 0); }
  asm volatile("" ::: "memory");
  asm volatile("" :: "v"(warm2));
  const char* sbase = myws + (size_t)(call & 1) * PAR_BYTES;

  const int m16 = lane & 15, g4 = lane >> 4, q4 = (lane & 15) >> 2, p4 = lane & 3;
  const unsigned xa0 = wv * 2048 + m16 * 64 + ((g4 ^ ((m16 >> 2) & 3)) * 16);
  const unsigned fwl = ((q4 >> 1) & 1) | ((g4 & 1) << 2);
  const unsigned wa_0 = wv * 4096 + g4 * 1024 + q4 * 128 + ((p4 ^ fwl) * 16) + ((p4 & 1) * 8);

  {
    Tile T;
    get_tile(rank, b, T);
    const bool xrem = T.xsrc != rank, wrem = T.owner != rank;
    const char* xg; unsigned xstride_c, xstride_ks;
    if (xrem) {
      xg = sbase + (size_t)T.mt * XT_BYTES + wv * 2048 + lane * 16; xstride_c = XS_BYTES; xstride_ks = 1024;
    } else {
      const int r16 = lane >> 2;
      const int c4 = (lane & 3) ^ ((r16 >> 2) & 3);
      xg = P.x + (size_t)(T.xt * 32 + r16) * (KDIM * 2) + wv * 64 + c4 * 16; xstride_c = KC * 2; xstride_ks = 16 * KDIM * 2;
    }
    const char* wg0; const char* wg1; unsigned wstride_c, wstride_k2;
    if (wrem) {
      wg0 = sbase + XSTG_BYTES + (size_t)T.wslot * WT_BYTES + wv * 4096 + lane * 16; wg1 = wg0 + 1024;
      wstride_c = WS_BYTES; wstride_k2 = 2048;
    } else {
      const int f0 = ((r8 >> 1) & 1), f1 = f0 | 4;
      wg0 = P.w + (size_t)(wv * 32 + r8) * (NS * 2) + T.ct * 128 + (((lane & 7) ^ f0) * 16);
      wg1 = P.w + (size_t)(wv * 32 + 8 + r8) * (NS * 2) + T.ct * 128 + (((lane & 7) ^ f1) * 16);
      wstride_c = KC * NS * 2; wstride_k2 = 16 * NS * 2;
    }
    f32x4 acc[2][2][2];
#pragma unroll
    for (int i = 0; i < 8; ++i) acc[i >> 2][(i >> 1) & 1][i & 1] = f32x4{0.f, 0.f, 0.f, 0.f};
    constexpr int cbase = 0;
    int issued = 0;
    for (int c = 0; c < NC; ++c) {
      int rdy = (int)lds_ld(ctl + 1) - cbase;
      while (issued <= c && rdy <= issued) {
        __builtin_amdgcn_s_sleep(CSLEEP); rdy = (int)lds_ld(ctl + 1) - cbase;
      }
      asm volatile("" ::: "memory");
      int lim = c + PFD;
      lim = lim < rdy ? lim : rdy;
      lim = lim < NC ? lim : NC;
      while (issued < lim) {
        const int sl = (cbase + issued) % NSLOT;
        const unsigned lw = lbase + LDS_W + sl * WS_BYTES + wv * 4096;
        const unsigned lx = lbase + LDS_X + sl * XS_BYTES + wv * 2048;
        const char* w0 = wg0 + (size_t)issued * wstride_c;
        const char* w1 = wg1 + (size_t)issued * wstride_c;
        const char* x0 = xg + (size_t)issued * xstride_c;
        {
        dma16<LD_AUX>(w0, lw);
        dma16<LD_AUX>(w1, lw + 1024);
        dma16<LD_AUX>(w0 + wstride_k2, lw + 2048);
        dma16<LD_AUX>(w1 + wstride_k2, lw + 3072);
        dma16<LD_AUX>(x0, lx);
        dma16<LD_AUX>(x0 + xstride_ks, lx + 1024);
        }
        ++issued;
      }
      switch (issued - c - 1) {
        case 0: WAITV(0); break;
        case 1: WAITV(6); break;
        case 2: WAITV(12); break;
        case 3: WAITV(18); break;
        default: WAITV(24); break;
      }
      {
        const int sl = c % NSLOT;
        const unsigned ax = lbase + LDS_X + sl * XS_BYTES + xa0;
        const unsigned aw = lbase + LDS_W + sl * WS_BYTES + wa_0;
        const unsigned aw00 = aw, aw01 = aw ^ 8, aw10 = aw ^ 64, aw11 = aw ^ 72;
        u32x4 xf[2]; u32x2 wf[2][2][2];
        asm volatile(
          "ds_read_b128 %0, %10\n\t"
          "ds_read_b64_tr_b16 %2, %11\n\t"
          "ds_read_b64_tr_b16 %3, %11 offset:512\n\t"
          "ds_read_b64_tr_b16 %4, %12\n\t"
          "ds_read_b64_tr_b16 %5, %12 offset:512\n\t"
          "ds_read_b128 %1, %10 offset:1024\n\t"
          "ds_read_b64_tr_b16 %6, %13\n\t"
          "ds_read_b64_tr_b16 %7, %13 offset:512\n\t"
          "ds_read_b64_tr_b16 %8, %14\n\t"
          "ds_read_b64_tr_b16 %9, %14 offset:512\n\t"
          "s_waitcnt lgkmcnt(0)"
          : "=&v"(xf[0]), "=&v"(xf[1]), "=&v"(wf[0][0][0]), "=&v"(wf[0][0][1]), "=&v"(wf[0][1][0]), "=&v"(wf[0][1][1]),
            "=&v"(wf[1][0][0]), "=&v"(wf[1][0][1]), "=&v"(wf[1][1][0]), "=&v"(wf[1][1][1])
          : "v"(ax), "v"(aw00), "v"(aw01), "v"(aw10), "v"(aw11) : "memory");
#pragma unroll
        for (int i = 0; i < 4; ++i) {
          const int n_ = i >> 1, t_ = i & 1;
          const u32x4 a = {wf[n_][t_][0][0], wf[n_][t_][0][1], wf[n_][t_][1][0], wf[n_][t_][1][1]};
          acc[0][n_][t_] = __builtin_amdgcn_mfma_f32_16x16x32_bf16(__builtin_bit_cast(bf16x8, a), __builtin_bit_cast(bf16x8, xf[0]), acc[0][n_][t_], 0, 0, 0);
          acc[1][n_][t_] = __builtin_amdgcn_mfma_f32_16x16x32_bf16(__builtin_bit_cast(bf16x8, a), __builtin_bit_cast(bf16x8, xf[1]), acc[1][n_][t_], 0, 0, 0);
        }
      }
    }
    f32x4 acc0, acc1;
    {
      f32x4* mine = (f32x4*)(smem + LDS_W + wv * 4096) + lane;
#pragma unroll
      for (int i = 0; i < 8; ++i) mine[(i >> 2) * (WS_BYTES / 16) + (i & 3) * 64] = acc[i >> 2][(i >> 1) & 1][i & 1];
      asm volatile("s_waitcnt lgkmcnt(0)" ::: "memory");
      if (lane == 0) __hip_atomic_fetch_add(ctl + 3, 1u, __ATOMIC_RELAXED, __HIP_MEMORY_SCOPE_WORKGROUP);
      while (lds_ld(ctl + 3) < 4u) { __builtin_amdgcn_s_sleep(0); }
      asm volatile("" ::: "memory");
      const int cg = lane & 7, srcl = (wv & 1) * 8 + (lane >> 3) + 16 * (cg & 3);
      const f32x4* src = (const f32x4*)(smem + LDS_W + wm * WS_BYTES + (cg >> 2) * 2048) + srcl;
      acc0 = src[0]; acc1 = src[64];
#pragma unroll
      for (int q = 1; q < 4; ++q) { acc0 += src[q * 256]; acc1 += src[q * 256 + 64]; }
    }
    const bool bad = lds_ld(ctl + 4) != 0;
    const f32x4 lo = (lane & 1) ? acc1 : acc0, hi = (lane & 1) ? acc0 : acc1;
    u32x4 o;
    o[0] = pack_relu(lo[0], lo[1]); o[1] = pack_relu(lo[2], lo[3]);
    o[2] = pack_relu(hi[0], hi[1]); o[3] = pack_relu(hi[2], hi[3]);
    if (bad) { o[0] = o[1] = o[2] = o[3] = 0x7fc07fc0u; }
    const int grow = T.xsrc * MS + T.xt * 32 + wv * 8 + (lane >> 3);
    char* op = P.out[T.owner] + (size_t)grow * (NS * 2) + T.ct * 128 + (lane & 7) * 16;
    if (!wrem) {
      st16_sys(op, o);
    } else {
      st16_sys(op, o);
      WAITV(0);
      if (lane == 0) st4_sys((unsigned*)(P.ws[T.owner] + FLAG_OFF) + FY_W + ((rank >> 1) * 32 + T.mt) * 4 + wv, call);
    }
    if (wv == 0) STAMP(4);
    asm volatile("agk_code_end0:");
  }
}

}

extern "C" void kernel_launch(int rank, int nranks, ncclComm_t comm,
                              hipStream_t stream,
                              void* const* d_in, const int* in_sizes, int n_in,
                              void* d_out, int out_size,
                              const int* in_dtypes, int out_dtype,
                              void* const* d_out_all,
                              void* const* d_ws_all, size_t ws_bytes) {
  if (nranks != NR || ws_bytes < WS_NEED || n_in < 2 || (unsigned)rank >= (unsigned)NR) {
    fprintf(stderr, "unsupported configuration\n");
    return;
  }
  Params p;
  p.x = (const char*)d_in[0];
  p.w = (const char*)d_in[1];
  for (int i = 0; i < NR; ++i) { p.out[i] = (char*)d_out_all[i]; p.ws[i] = (char*)d_ws_all[i]; }
  for (int i = 0; i < NR; ++i) p.wsr[i] = (char*)d_ws_all[(rank + i) & 7];
  p.myws = (char*)d_ws_all[rank];
  p.rank = rank;
  ag_gemm_kernel<<<NWG, NTHREADS, 0, stream>>>(p);
  HIP_CHECK(hipGetLastError());
}
```

```cpp
#include <hip/hip_runtime.h>
#include <rccl/rccl.h>
#include <cstdio>
#include <type_traits>

#define HIP_CHECK(expr) do { hipError_t _e = (expr); if (_e != hipSuccess) { \
    fprintf(stderr, "HIP error %s at %s:%d\n", hipGetErrorString(_e), __FILE__, __LINE__); \
    return; } } while (0)

#define MODE_AG 0
#define ACQ_MODE 2
#define PFD 2
#define NPW 1
#define NP2 64
#define NPUSHWG 256
#define WT_OUT 1
#define PHASES 0
#define XMAP 5
#define WARM 1
#define AFF 1
#define RAWP 1
#define SUBW 2
#define TAILX 0
#define CSLEEP 0
#define HALFU 0
#define FSPREAD 2
#define CS 0
#define CNTI(i) (CNT_W + (i))
#define WARMC 1
#define LD_AUX 0

namespace {

constexpr int NR = 8;
constexpr int KDIM = 2048;
constexpr int MS = 256, NS = 256;
constexpr int BM = 32, BN = 64, KC = 128, NC = KDIM / KC;
constexpr int NWG = 256;
constexpr int NWAVES = 5 + NPW;
constexpr int NTHREADS = NWAVES * 64;

constexpr int NMT = 32;
constexpr int PPC = 70 * (HALFU + 1);
constexpr int NPAIRS = PPC * NC;
constexpr int NP = NPUSHWG + (NPW - 1) * NP2;
static_assert(NPAIRS >= 2 * NP, "two static units per pusher wave");

constexpr size_t XT_BYTES = (size_t)BM * KDIM * 2;
constexpr size_t WT_BYTES = (size_t)BN * KDIM * 2;
constexpr size_t XSTG_BYTES = (size_t)NMT * XT_BYTES;
constexpr size_t WSTG_BYTES = MODE_AG ? 0 : 4 * WT_BYTES;
constexpr size_t PAR_BYTES = XSTG_BYTES + WSTG_BYTES;
constexpr size_t FLAG_OFF = 16u << 20;
static_assert(2 * PAR_BYTES <= FLAG_OFF, "staging overflows");
constexpr size_t WS_NEED = FLAG_OFF + (256u << 10);
constexpr int FX_W = 0;
constexpr int FW_W = 2048;
constexpr int FY_W = 4096;
#define FS 32
constexpr int FXS_W = 16384, FWS_W = FXS_W + 512 * FS, FYS_W = FWS_W + 64 * FS;
static_assert(FYS_W + 128 * FS <= 65536, "flag area overflows");
constexpr int CNT_W = 8192;
constexpr int TMO_W = 8192 + 1024;
constexpr int TICKET_W = 8192 + 2048;
constexpr int DBG_W = 40960;

constexpr unsigned long long DEADLINE = 50000000ull;

constexpr int XS_BYTES = BM * KC * 2;
constexpr int WS_BYTES = BN * KC * 2;
constexpr int NSLOT = 6;
constexpr int LDS_X = 0;
constexpr int LDS_W = NSLOT * XS_BYTES;
constexpr int LDS_CTL = LDS_W + NSLOT * WS_BYTES;
constexpr int LDS_BYTES = LDS_CTL + 64;

typedef __attribute__((ext_vector_type(4))) unsigned u32x4;
typedef __attribute__((ext_vector_type(2))) unsigned u32x2;
typedef __attribute__((ext_vector_type(8))) short bf16x8;
typedef __attribute__((ext_vector_type(4))) float f32x4;
typedef __attribute__((ext_vector_type(2))) float f32x2;
typedef __attribute__((ext_vector_type(2))) __bf16 bf16x2;
typedef __attribute__((address_space(3))) unsigned lds_u32;

struct Params {
  const char* x;
  const char* w;
  char* myws;
  int rank;
  char* wsr[NR];
  char* out[NR];
  char* ws[NR];
};

__device__ __forceinline__ void st16_sys(void* p, u32x4 v) {
  asm volatile("global_store_dwordx4 %0, %1, off sc0 sc1\n\ts_nop 1" :: "v"(p), "v"(v) : "memory");
}
__device__ __forceinline__ void st4_sys(void* p, unsigned v) {
  asm volatile("global_store_dword %0, %1, off sc0 sc1" :: "v"(p), "v"(v) : "memory");
}
__device__ __forceinline__ unsigned ld4_sys(const void* p) {
  return __hip_atomic_load((const unsigned*)p, __ATOMIC_RELAXED, __HIP_MEMORY_SCOPE_SYSTEM);
}
__device__ __forceinline__ unsigned lds_ld(unsigned* p) {
  return __hip_atomic_load(p, __ATOMIC_RELAXED, __HIP_MEMORY_SCOPE_WORKGROUP);
}
__device__ __forceinline__ void lds_st(unsigned* p, unsigned v) {
  __hip_atomic_store(p, v, __ATOMIC_RELAXED, __HIP_MEMORY_SCOPE_WORKGROUP);
}
#define WAITV(n) asm volatile("s_waitcnt vmcnt(" #n ")" ::: "memory")

template <int AUX>
__device__ __forceinline__ void dma16(const char* g, unsigned l) {
  if constexpr (AUX == 0)
    asm volatile("s_mov_b32 m0, %1\n\ts_nop 0\n\tglobal_load_lds_dwordx4 %0, off" :: "v"(g), "s"(l) : "memory");
  else if constexpr (AUX == 16)
    asm volatile("s_mov_b32 m0, %1\n\ts_nop 0\n\tglobal_load_lds_dwordx4 %0, off sc1" :: "v"(g), "s"(l) : "memory");
  else if constexpr (AUX == 17)
    asm volatile("s_mov_b32 m0, %1\n\ts_nop 0\n\tglobal_load_lds_dwordx4 %0, off sc0 sc1" :: "v"(g), "s"(l) : "memory");
  else
    asm volatile("s_mov_b32 m0, %1\n\ts_nop 0\n\tglobal_load_lds_dwordx4 %0, off nt" :: "v"(g), "s"(l) : "memory");
}

__device__ __forceinline__ unsigned pack_relu(float a, float b) {
  a = a < 0.f ? 0.f : a;
  b = b < 0.f ? 0.f : b;
  f32x2 f = {a, b};
  bf16x2 h = __builtin_convertvector(f, bf16x2);
  return __builtin_bit_cast(unsigned, h);
}

#define STAMP(idx) do {} while (0)

struct Tile { int mt, xsrc, xt, owner, ct, wslot; };

__device__ __forceinline__ void get_tile(int rank, int b, Tile& T) {
  const int xcd = b & 7, s = b >> 3;
  const int nt = ((s >> 3) & 1) + 2 * (xcd >> 2) + 4 * (s >> 4);
  const int mt = (xcd & 3) + 4 * (s & 7);
  const int par_me = rank & 1;
  T.wslot = nt - 4;
  T.owner = nt < 4 ? rank : (1 - par_me) + 2 * (nt - 4);
  T.ct = nt < 4 ? nt : (rank >> 1);
  T.mt = mt;
  if (mt < 20) { T.xsrc = par_me + 2 * (mt / 5); T.xt = mt % 5; }
  else { T.xsrc = (1 - par_me) + 2 * ((mt - 20) / 3); T.xt = 5 + (mt - 20) % 3; }
}

__global__ void __launch_bounds__(NTHREADS, 1) ag_gemm_kernel(Params P) {
  __shared__ __attribute__((aligned(1024))) char smem[LDS_BYTES];
  unsigned* ctl = (unsigned*)(smem + LDS_CTL);
  const int tid = threadIdx.x, lane = tid & 63;
  const int wv = __builtin_amdgcn_readfirstlane(tid >> 6);
  const int b = blockIdx.x;
  const int rank = P.rank;
  char* myws = P.myws;
  unsigned* myf = (unsigned*)(myws + FLAG_OFF);
  unsigned* tmo = myf + TMO_W;

  if (wv == 0) STAMP(0);
  const char* code_pc; int code_len;
  asm volatile("agk_code_begin:\n\ts_getpc_b64 %0\n\ts_mov_b32 %1, agk_code_end0-agk_code_begin-8\n\t"
               "s_max_i32 %1, %1, agk_code_end1-agk_code_begin-8\n\ts_max_i32 %1, %1, agk_code_end2-agk_code_begin-8"
               : "=s"(code_pc), "=s"(code_len) :: "scc");

  if (wv >= 5) {
    const int g = wv == 5 ? b : NPUSHWG + b;
    __builtin_amdgcn_s_barrier();
    if (b >= (wv == 5 ? NPUSHWG : NP2)) return;
    unsigned call;
    asm volatile("global_load_dword %0, %1, off sc0 sc1" : "=v"(call) : "v"(myf + CNTI(NWG + g)) : "memory");
    const int r = lane >> 3;
    const int r16 = lane >> 2;
    const unsigned xl = r16 * (KDIM * 2) + ((lane & 3) ^ ((r16 >> 2) & 3)) * 16;
    const unsigned wl0 = r * (NS * 2) + ((lane & 7) ^ ((r >> 1) & 1)) * 16;
    const unsigned wl1 = (r + 8) * (NS * 2) + ((lane & 7) ^ (((r >> 1) & 1) | 4)) * 16;
    const int dsel = g % 7;
    struct Unit { u32x4 v[4]; char* dp; unsigned* fp; int idx; };
    auto fetch = [&](int i, Unit& U) {
      const int c = i / (PPC / 7);
      const int d = 1 + dsel, u = i % (PPC / 7);
      const int dest = (rank + d) & 7;
      char* dws = P.wsr[d];
      bool isw = false; int xu = u;
      if (d & 1) { if (u < 4) isw = true; else xu = 10 + (u - 4); }
      const int t = xu >> 1, h = xu & 1;
      const int dmt = t < 5 ? (rank >> 1) * 5 + t : 20 + (rank >> 1) * 3 + (t - 5);
      const char* sb = isw ? P.w + (size_t)(c * KC + u * 32) * (NS * 2) + (dest >> 1) * 128
                           : P.x + (size_t)(t * 32) * (KDIM * 2) + c * (KC * 2) + h * 128;
      const unsigned o0 = isw ? wl0 : xl, o1 = isw ? wl1 : xl + 16 * KDIM * 2;
      const unsigned o2 = isw ? wl0 + 16 * NS * 2 : xl + 64, o3 = isw ? wl1 + 16 * NS * 2 : xl + 64 + 16 * KDIM * 2;
      asm volatile("global_load_dwordx4 %0, %1, off" : "=v"(U.v[0]) : "v"(sb + o0) : "memory");
      asm volatile("global_load_dwordx4 %0, %1, off" : "=v"(U.v[1]) : "v"(sb + o1) : "memory");
      asm volatile("global_load_dwordx4 %0, %1, off" : "=v"(U.v[2]) : "v"(sb + o2) : "memory");
      asm volatile("global_load_dwordx4 %0, %1, off" : "=v"(U.v[3]) : "v"(sb + o3) : "memory");
      U.idx = i;
      U.dp = dws + (isw ? (unsigned)(XSTG_BYTES + (rank >> 1) * WT_BYTES) + c * WS_BYTES + u * 4096
                        : (unsigned)(dmt * XT_BYTES) + c * XS_BYTES + h * 4096) + lane * 16;
      U.fp = (unsigned*)(dws + FLAG_OFF) + (isw ? FWS_W + ((rank >> 1) * NC + c) * FS + u : FXS_W + (dmt * NC + c) * FS + h);
    };
    Unit UA, UB;
    const int g0 = g / 7, NPL = NP / 7 + (g % 7 < NP % 7 ? 1 : 0), NPAIRSL = NPAIRS / 7;
    unsigned* ticket = myf + TICKET_W + dsel * 64;
    fetch(g0, UA);
    fetch(g0 + NPL, UB);
    WAITV(4);
    asm volatile("" : "+v"(call));
    call = __builtin_amdgcn_readfirstlane(call) + 1;
    if (lane == 0) st4_sys(myf + CNTI(NWG + g), call);
    const size_t poff = (size_t)(call & 1) * PAR_BYTES;
    const unsigned tbase = (call - 1) * (unsigned)(NPAIRSL - NPL) - 2u * NPL;
    STAMP(6);
    int nxt = g0 + NPL;
    auto step = [&](Unit& U) -> bool {
      char* dp = U.dp + poff;
#pragma unroll
      for (int ks = 0; ks < (HALFU ? 2 : 4); ++ks) {
        st16_sys(dp + ks * 1024, U.v[ks]);
        if ((ks == 1 || ks == 2) && (TAILX == 0 || U.idx < NPAIRSL - TAILX)) WAITV(1);
      }
      unsigned tk = 0;
      const bool more = nxt < NPAIRSL;
      if (more && lane == 0) tk = __hip_atomic_fetch_add(ticket, 1u, __ATOMIC_RELAXED, __HIP_MEMORY_SCOPE_SYSTEM);
      WAITV(0);
      if (lane == 0) st4_sys(U.fp, call);
      if (!more) return false;
      nxt = (int)(__builtin_amdgcn_readfirstlane(tk) - tbase);
      if (nxt < NPAIRSL) fetch(nxt, U);
      return true;
    };
    while (true) {
      if (!step(UA)) break;
      if (!step(UB)) break;
    }
    STAMP(1);
    asm volatile("agk_code_end1:");
    return;
  }

  if (wv == 4) {
    unsigned call = 0;
    if (lane < 8) lds_st(ctl + lane, 0u);
    asm volatile("s_waitcnt lgkmcnt(0)" ::: "memory");
    __builtin_amdgcn_s_barrier();
    unsigned warmv = 0;
    {
      Tile T;
      get_tile(rank, b, T);
      if (lane == 1) warmv = ld4_sys(P.out[T.owner] + (size_t)(T.xsrc * MS + T.xt * 32) * (NS * 2) + T.ct * 128);
      if (lane == 2) warmv = ld4_sys((unsigned*)(P.ws[T.owner] + FLAG_OFF) + FYS_W + ((rank >> 1) * 32 + T.mt) * FS);
    }
    if (lane == 0) {
      call = ld4_sys(myf + CNTI(b)) + 1;
      st4_sys(myf + CNTI(b), call);
    }
    call = __builtin_amdgcn_readfirstlane(call);
    __builtin_amdgcn_fence(__ATOMIC_ACQUIRE, "");
    asm volatile("s_waitcnt vmcnt(0)" ::: "memory");
    if (lane == 0) lds_st(ctl + 0, call);
    asm volatile("" :: "v"(warmv));
    const unsigned long long t0 = __builtin_amdgcn_s_memrealtime();
    bool bad = false;
    {
      Tile T;
      get_tile(rank, b, T);
      const bool xrem = T.xsrc != rank, wrem = T.owner != rank;
      int rdy = 0;
      if (!xrem && !wrem) { rdy = NC; if (lane == 0) lds_st(ctl + 1, (unsigned)NC); }
      unsigned it = 0;
      while (rdy < NC) {
        unsigned f = call;
        const int cx = rdy + (lane >> 1), cw = rdy + ((lane - 8) >> 2);
        if (lane < 8) { if (xrem && cx < NC) f = ld4_sys(myf + FXS_W + (T.mt * NC + cx) * FS + (lane & 1)); }
        else if (lane < 24) { if (wrem && cw < NC) f = ld4_sys(myf + FWS_W + (T.wslot * NC + cw) * FS + (lane & 3)); }
        unsigned tm = 0;
        if ((++it & 15) == 0) tm = __hip_atomic_load(tmo, __ATOMIC_RELAXED, __HIP_MEMORY_SCOPE_SYSTEM);
        const unsigned bl = (unsigned)__ballot(f >= call);
        unsigned bx = bl & 0xffu, bw = (bl >> 8) & 0xffffu;
        bx &= bx >> 1; bx &= 0x55u;
        bw &= bw >> 1; bw &= bw >> 2; bw &= 0x1111u;
        const int nrx = __builtin_ctz((~bx & 0x55u) | 0x100u) >> 1;
        const int nrw = __builtin_ctz((~bw & 0x1111u) | 0x10000u) >> 2;
        int nr = rdy + (nrx < nrw ? nrx : nrw);
        nr = nr < NC ? nr : NC;
        if (tm == call || ((it & 15) == 0 && __builtin_amdgcn_s_memrealtime() - t0 > DEADLINE)) {
          if (lane == 0) { __hip_atomic_store(tmo, call, __ATOMIC_RELAXED, __HIP_MEMORY_SCOPE_SYSTEM); lds_st(ctl + 4, 1u); }
          bad = true; nr = NC;
        }
        if (nr > rdy) {
          if (lane == 0) lds_st(ctl + 1, (unsigned)nr);
          rdy = nr;
        } else {
          __builtin_amdgcn_s_sleep(1);
        }
      }
    }
    STAMP(2);
    if (b < 8) {
      bool done = false;
      while (!done) {
        const unsigned fy = ld4_sys(myf + FYS_W + (b * 16 + (lane >> 2)) * FS + (lane & 3));
        const unsigned tm = __hip_atomic_load(tmo, __ATOMIC_RELAXED, __HIP_MEMORY_SCOPE_SYSTEM);
        done = __all(fy >= call);
        if (!done) {
          if (tm == call || __builtin_amdgcn_s_memrealtime() - t0 > DEADLINE) {
            if (lane == 0) __hip_atomic_store(tmo, call, __ATOMIC_RELAXED, __HIP_MEMORY_SCOPE_SYSTEM);
            bad = true; done = true;
          } else __builtin_amdgcn_s_sleep(1);
        }
      }
      if (bad && lane == 0) {
        ((unsigned short*)P.out[rank])[(size_t)(rank * MS) * NS + b] = 0x7fc0;
      }
      STAMP(3);
    }
    asm volatile("agk_code_end2:");
    return;
  }

  const int wm = wv >> 1, wn = wv & 1;
  const unsigned lbase = (unsigned)(size_t)(__attribute__((address_space(3))) char*)smem;
  const int r8 = lane >> 3;
  __builtin_amdgcn_s_barrier();
  asm volatile("" ::: "memory");
  unsigned warm2 = 0;
  if ((int)(tid * 64u) < code_len) warm2 = *(const unsigned*)(code_pc + tid * 64u);
  unsigned call = lds_ld(ctl + 0);
  while (call == 0) { __builtin_amdgcn_s_sleep(0); call = lds_ld(ctl + 0); }
  asm volatile("" ::: "memory");
  asm volatile("" :: "v"(warm2));
  const char* sbase = myws + (size_t)(call & 1) * PAR_BYTES;

  const int m16 = lane & 15, g4 = lane >> 4, q4 = (lane & 15) >> 2, p4 = lane & 3;
  const unsigned xa0 = wv * 2048 + m16 * 64 + ((g4 ^ ((m16 >> 2) & 3)) * 16);
  const unsigned fwl = ((q4 >> 1) & 1) | ((g4 & 1) << 2);
  const unsigned wa_0 = wv * 4096 + g4 * 1024 + q4 * 128 + ((p4 ^ fwl) * 16) + ((p4 & 1) * 8);

  {
    Tile T;
    get_tile(rank, b, T);
    const bool xrem = T.xsrc != rank, wrem = T.owner != rank;
    const char* xg; unsigned xstride_c, xstride_ks;
    if (xrem) {
      xg = sbase + (size_t)T.mt * XT_BYTES + wv * 2048 + lane * 16; xstride_c = XS_BYTES; xstride_ks = 1024;
    } else {
      const int r16 = lane >> 2;
      const int c4 = (lane & 3) ^ ((r16 >> 2) & 3);
      xg = P.x + (size_t)(T.xt * 32 + r16) * (KDIM * 2) + wv * 64 + c4 * 16; xstride_c = KC * 2; xstride_ks = 16 * KDIM * 2;
    }
    const char* wg0; const char* wg1; unsigned wstride_c, wstride_k2;
    if (wrem) {
      wg0 = sbase + XSTG_BYTES + (size_t)T.wslot * WT_BYTES + wv * 4096 + lane * 16; wg1 = wg0 + 1024;
      wstride_c = WS_BYTES; wstride_k2 = 2048;
    } else {
      const int f0 = ((r8 >> 1) & 1), f1 = f0 | 4;
      wg0 = P.w + (size_t)(wv * 32 + r8) * (NS * 2) + T.ct * 128 + (((lane & 7) ^ f0) * 16);
      wg1 = P.w + (size_t)(wv * 32 + 8 + r8) * (NS * 2) + T.ct * 128 + (((lane & 7) ^ f1) * 16);
      wstride_c = KC * NS * 2; wstride_k2 = 16 * NS * 2;
    }
    f32x4 acc[2][2][2];
#pragma unroll
    for (int i = 0; i < 8; ++i) acc[i >> 2][(i >> 1) & 1][i & 1] = f32x4{0.f, 0.f, 0.f, 0.f};
    constexpr int cbase = 0;
    int issued = 0;
    for (int c = 0; c < NC; ++c) {
      int rdy = (int)lds_ld(ctl + 1) - cbase;
      while (issued <= c && rdy <= issued) {
        __builtin_amdgcn_s_sleep(CSLEEP); rdy = (int)lds_ld(ctl + 1) - cbase;
      }
      asm volatile("" ::: "memory");
      int lim = c + PFD;
      lim = lim < rdy ? lim : rdy;
      lim = lim < NC ? lim : NC;
      while (issued < lim) {
        const int sl = (cbase + issued) % NSLOT;
        const unsigned lw = lbase + LDS_W + sl * WS_BYTES + wv * 4096;
        const unsigned lx = lbase + LDS_X + sl * XS_BYTES + wv * 2048;
        const char* w0 = wg0 + (size_t)issued * wstride_c;
        const char* w1 = wg1 + (size_t)issued * wstride_c;
        const char* x0 = xg + (size_t)issued * xstride_c;
        {
        dma16<LD_AUX>(w0, lw);
        dma16<LD_AUX>(w1, lw + 1024);
        dma16<LD_AUX>(w0 + wstride_k2, lw + 2048);
        dma16<LD_AUX>(w1 + wstride_k2, lw + 3072);
        dma16<LD_AUX>(x0, lx);
        dma16<LD_AUX>(x0 + xstride_ks, lx + 1024);
        }
        ++issued;
      }
      switch (issued - c - 1) {
        case 0: WAITV(0); break;
        case 1: WAITV(6); break;
        case 2: WAITV(12); break;
        case 3: WAITV(18); break;
        default: WAITV(24); break;
      }
      {
        const int sl = c % NSLOT;
        const unsigned ax = lbase + LDS_X + sl * XS_BYTES + xa0;
        const unsigned aw = lbase + LDS_W + sl * WS_BYTES + wa_0;
        const unsigned aw00 = aw, aw01 = aw ^ 8, aw10 = aw ^ 64, aw11 = aw ^ 72;
        u32x4 xf[2]; u32x2 wf[2][2][2];
        asm volatile(
          "ds_read_b128 %0, %10\n\t"
          "ds_read_b64_tr_b16 %2, %11\n\t"
          "ds_read_b64_tr_b16 %3, %11 offset:512\n\t"
          "ds_read_b64_tr_b16 %4, %12\n\t"
          "ds_read_b64_tr_b16 %5, %12 offset:512\n\t"
          "ds_read_b128 %1, %10 offset:1024\n\t"
          "ds_read_b64_tr_b16 %6, %13\n\t"
          "ds_read_b64_tr_b16 %7, %13 offset:512\n\t"
          "ds_read_b64_tr_b16 %8, %14\n\t"
          "ds_read_b64_tr_b16 %9, %14 offset:512\n\t"
          "s_waitcnt lgkmcnt(0)"
          : "=&v"(xf[0]), "=&v"(xf[1]), "=&v"(wf[0][0][0]), "=&v"(wf[0][0][1]), "=&v"(wf[0][1][0]), "=&v"(wf[0][1][1]),
            "=&v"(wf[1][0][0]), "=&v"(wf[1][0][1]), "=&v"(wf[1][1][0]), "=&v"(wf[1][1][1])
          : "v"(ax), "v"(aw00), "v"(aw01), "v"(aw10), "v"(aw11) : "memory");
#pragma unroll
        for (int i = 0; i < 4; ++i) {
          const int n_ = i >> 1, t_ = i & 1;
          const u32x4 a = {wf[n_][t_][0][0], wf[n_][t_][0][1], wf[n_][t_][1][0], wf[n_][t_][1][1]};
          acc[0][n_][t_] = __builtin_amdgcn_mfma_f32_16x16x32_bf16(__builtin_bit_cast(bf16x8, a), __builtin_bit_cast(bf16x8, xf[0]), acc[0][n_][t_], 0, 0, 0);
          acc[1][n_][t_] = __builtin_amdgcn_mfma_f32_16x16x32_bf16(__builtin_bit_cast(bf16x8, a), __builtin_bit_cast(bf16x8, xf[1]), acc[1][n_][t_], 0, 0, 0);
        }
      }
    }
    f32x4 acc0, acc1;
    {
      f32x4* mine = (f32x4*)(smem + LDS_W + wv * 4096) + lane;
#pragma unroll
      for (int i = 0; i < 8; ++i) mine[(i >> 2) * (WS_BYTES / 16) + (i & 3) * 64] = acc[i >> 2][(i >> 1) & 1][i & 1];
      asm volatile("s_waitcnt lgkmcnt(0)" ::: "memory");
      if (lane == 0) __hip_atomic_fetch_add(ctl + 3, 1u, __ATOMIC_RELAXED, __HIP_MEMORY_SCOPE_WORKGROUP);
      while (lds_ld(ctl + 3) < 4u) { __builtin_amdgcn_s_sleep(0); }
      asm volatile("" ::: "memory");
      const int cg = lane & 7, srcl = (wv & 1) * 8 + (lane >> 3) + 16 * (cg & 3);
      const f32x4* src = (const f32x4*)(smem + LDS_W + wm * WS_BYTES + (cg >> 2) * 2048) + srcl;
      acc0 = src[0]; acc1 = src[64];
#pragma unroll
      for (int q = 1; q < 4; ++q) { acc0 += src[q * 256]; acc1 += src[q * 256 + 64]; }
    }
    const bool bad = lds_ld(ctl + 4) != 0;
    const f32x4 lo = (lane & 1) ? acc1 : acc0, hi = (lane & 1) ? acc0 : acc1;
    u32x4 o;
    o[0] = pack_relu(lo[0], lo[1]); o[1] = pack_relu(lo[2], lo[3]);
    o[2] = pack_relu(hi[0], hi[1]); o[3] = pack_relu(hi[2], hi[3]);
    if (bad) { o[0] = o[1] = o[2] = o[3] = 0x7fc07fc0u; }
    const int grow = T.xsrc * MS + T.xt * 32 + wv * 8 + (lane >> 3);
    char* op = P.out[T.owner] + (size_t)grow * (NS * 2) + T.ct * 128 + (lane & 7) * 16;
    if (!wrem) {
      st16_sys(op, o);
    } else {
      st16_sys(op, o);
      WAITV(0);
      if (lane == 0) st4_sys((unsigned*)(P.ws[T.owner] + FLAG_OFF) + FYS_W + ((rank >> 1) * 32 + T.mt) * FS + wv, call);
    }
    if (wv == 0) STAMP(4);
    asm volatile("agk_code_end0:");
  }
}

}

extern "C" void kernel_launch(int rank, int nranks, ncclComm_t comm,
                              hipStream_t stream,
                              void* const* d_in, const int* in_sizes, int n_in,
                              void* d_out, int out_size,
                              const int* in_dtypes, int out_dtype,
                              void* const* d_out_all,
                              void* const* d_ws_all, size_t ws_bytes) {
  if (nranks != NR || ws_bytes < WS_NEED || n_in < 2 || (unsigned)rank >= (unsigned)NR) {
    fprintf(stderr, "unsupported configuration\n");
    return;
  }
  Params p;
  p.x = (const char*)d_in[0];
  p.w = (const char*)d_in[1];
  for (int i = 0; i < NR; ++i) { p.out[i] = (char*)d_out_all[i]; p.ws[i] = (char*)d_ws_all[i]; }
  for (int i = 0; i < NR; ++i) p.wsr[i] = (char*)d_ws_all[(rank + i) & 7];
  p.myws = (char*)d_ws_all[rank];
  p.rank = rank;
  ag_gemm_kernel<<<NWG, NTHREADS, 0, stream>>>(p);
  HIP_CHECK(hipGetLastError());
}
```

```cpp
#include <hip/hip_runtime.h>
#include <rccl/rccl.h>
#include <cstdio>
#include <type_traits>

#define HIP_CHECK(expr) do { hipError_t _e = (expr); if (_e != hipSuccess) { \
    fprintf(stderr, "HIP error %s at %s:%d\n", hipGetErrorString(_e), __FILE__, __LINE__); \
    return; } } while (0)

#define MODE_AG 0
#define ACQ_MODE 2
#define PFD 2
#define NPW 1
#define NP2 64
#define NPUSHWG 256
#define WT_OUT 1
#define PHASES 0
#define XMAP 5
#define WARM 1
#define AFF 1
#define RAWP 1
#define SUBW 2
#define TAILX 0
#define CSLEEP 0
#define HALFU 0
#define FSPREAD 0
#define CS 0
#define CNTI(i) (CNT_W + (i))
#define WARMC 1
#define LD_AUX 0

namespace {

constexpr int NR = 8;
constexpr int KDIM = 2048;
constexpr int MS = 256, NS = 256;
constexpr int BM = 32, BN = 64, KC = 128, NC = KDIM / KC;
constexpr int NWG = 256;
constexpr int NWAVES = 5 + NPW;
constexpr int NTHREADS = NWAVES * 64;

constexpr int NMT = 32;
constexpr int PPC = 70 * (HALFU + 1);
constexpr int NPAIRS = PPC * NC;
constexpr int NP = NPUSHWG + (NPW - 1) * NP2;
static_assert(NPAIRS >= 2 * NP, "two static units per pusher wave");

constexpr size_t XT_BYTES = (size_t)BM * KDIM * 2;
constexpr size_t WT_BYTES = (size_t)BN * KDIM * 2;
constexpr size_t XSTG_BYTES = (size_t)NMT * XT_BYTES;
constexpr size_t WSTG_BYTES = MODE_AG ? 0 : 4 * WT_BYTES;
constexpr size_t PAR_BYTES = XSTG_BYTES + WSTG_BYTES;
constexpr size_t FLAG_OFF = 16u << 20;
static_assert(2 * PAR_BYTES <= FLAG_OFF, "staging overflows");
constexpr size_t WS_NEED = FLAG_OFF + (256u << 10);
constexpr int FX_W = 0;
constexpr int FW_W = 2048;
constexpr int FY_W = 4096;
#define FS 32
constexpr int FXS_W = 16384, FWS_W = FXS_W + 512 * FS, FYS_W = FWS_W + 64 * FS;
static_assert(FYS_W + 128 * FS <= 65536, "flag area overflows");
constexpr int CNT_W = 8192;
constexpr int TMO_W = 8192 + 1024;
constexpr int TICKET_W = 8192 + 2048;
constexpr int DBG_W = 40960;

constexpr unsigned long long DEADLINE = 50000000ull;

constexpr int XS_BYTES = BM * KC * 2;
constexpr int WS_BYTES = BN * KC * 2;
constexpr int NSLOT = 6;
constexpr int LDS_X = 0;
constexpr int LDS_W = NSLOT * XS_BYTES;
constexpr int LDS_CTL = LDS_W + NSLOT * WS_BYTES;
constexpr int LDS_BYTES = LDS_CTL + 64;

typedef __attribute__((ext_vector_type(4))) unsigned u32x4;
typedef __attribute__((ext_vector_type(2))) unsigned u32x2;
typedef __attribute__((ext_vector_type(8))) short bf16x8;
typedef __attribute__((ext_vector_type(4))) float f32x4;
typedef __attribute__((ext_vector_type(2))) float f32x2;
typedef __attribute__((ext_vector_type(2))) __bf16 bf16x2;
typedef __attribute__((address_space(3))) unsigned lds_u32;

struct Params {
  const char* x;
  const char* w;
  char* myws;
  int rank;
  char* wsr[NR];
  char* out[NR];
  char* ws[NR];
};

__device__ __forceinline__ void st16_sys(void* p, u32x4 v) {
  asm volatile("global_store_dwordx4 %0, %1, off sc0 sc1\n\ts_nop 1" :: "v"(p), "v"(v) : "memory");
}
__device__ __forceinline__ void st4_sys(void* p, unsigned v) {
  asm volatile("global_store_dword %0, %1, off sc0 sc1" :: "v"(p), "v"(v) : "memory");
}
__device__ __forceinline__ unsigned ld4_sys(const void* p) {
  return __hip_atomic_load((const unsigned*)p, __ATOMIC_RELAXED, __HIP_MEMORY_SCOPE_SYSTEM);
}
__device__ __forceinline__ unsigned lds_ld(unsigned* p) {
  return __hip_atomic_load(p, __ATOMIC_RELAXED, __HIP_MEMORY_SCOPE_WORKGROUP);
}
__device__ __forceinline__ void lds_st(unsigned* p, unsigned v) {
  __hip_atomic_store(p, v, __ATOMIC_RELAXED, __HIP_MEMORY_SCOPE_WORKGROUP);
}
#define WAITV(n) asm volatile("s_waitcnt vmcnt(" #n ")" ::: "memory")

template <int AUX>
__device__ __forceinline__ void dma16(const char* g, unsigned l) {
  if constexpr (AUX == 0)
    asm volatile("s_mov_b32 m0, %1\n\ts_nop 0\n\tglobal_load_lds_dwordx4 %0, off" :: "v"(g), "s"(l) : "memory");
  else if constexpr (AUX == 16)
    asm volatile("s_mov_b32 m0, %1\n\ts_nop 0\n\tglobal_load_lds_dwordx4 %0, off sc1" :: "v"(g), "s"(l) : "memory");
  else if constexpr (AUX == 17)
    asm volatile("s_mov_b32 m0, %1\n\ts_nop 0\n\tglobal_load_lds_dwordx4 %0, off sc0 sc1" :: "v"(g), "s"(l) : "memory");
  else
    asm volatile("s_mov_b32 m0, %1\n\ts_nop 0\n\tglobal_load_lds_dwordx4 %0, off nt" :: "v"(g), "s"(l) : "memory");
}

__device__ __forceinline__ unsigned pack_relu(float a, float b) {
  a = a < 0.f ? 0.f : a;
  b = b < 0.f ? 0.f : b;
  f32x2 f = {a, b};
  bf16x2 h = __builtin_convertvector(f, bf16x2);
  return __builtin_bit_cast(unsigned, h);
}

#define STAMP(idx) do {} while (0)

struct Tile { int mt, xsrc, xt, owner, ct, wslot; };

__device__ __forceinline__ void get_tile(int rank, int b, Tile& T) {
  const int xcd = b & 7, s = b >> 3;
  const int nt = ((s >> 3) & 1) + 2 * (xcd >> 2) + 4 * (s >> 4);
  const int mt = (xcd & 3) + 4 * (s & 7);
  const int par_me = rank & 1;
  T.wslot = nt - 4;
  T.owner = nt < 4 ? rank : (1 - par_me) + 2 * (nt - 4);
  T.ct = nt < 4 ? nt : (rank >> 1);
  T.mt = mt;
  if (mt < 20) { T.xsrc = par_me + 2 * (mt / 5); T.xt = mt % 5; }
  else { T.xsrc = (1 - par_me) + 2 * ((mt - 20) / 3); T.xt = 5 + (mt - 20) % 3; }
}

__global__ void __launch_bounds__(NTHREADS, 1) ag_gemm_kernel(Params P) {
  __shared__ __attribute__((aligned(1024))) char smem[LDS_BYTES];
  unsigned* ctl = (unsigned*)(smem + LDS_CTL);
  const int tid = threadIdx.x, lane = tid & 63;
  const int wv = __builtin_amdgcn_readfirstlane(tid >> 6);
  const int b = blockIdx.x;
  const int rank = P.rank;
  char* myws = P.myws;
  unsigned* myf = (unsigned*)(myws + FLAG_OFF);
  unsigned* tmo = myf + TMO_W;

  if (wv == 0) STAMP(0);
  const char* code_pc; int code_len;
  asm volatile("agk_code_begin:\n\ts_getpc_b64 %0\n\ts_mov_b32 %1, agk_code_end0-agk_code_begin-8\n\t"
               "s_max_i32 %1, %1, agk_code_end1-agk_code_begin-8\n\ts_max_i32 %1, %1, agk_code_end2-agk_code_begin-8"
               : "=s"(code_pc), "=s"(code_len) :: "scc");

  if (wv >= 5) {
    const int g = wv == 5 ? b : NPUSHWG + b;
    __builtin_amdgcn_s_barrier();
    if (b >= (wv == 5 ? NPUSHWG : NP2)) return;
    unsigned call;
    asm volatile("global_load_dword %0, %1, off sc0 sc1" : "=v"(call) : "v"(myf + CNTI(NWG + g)) : "memory");
    const int r = lane >> 3;
    const int r16 = lane >> 2;
    const unsigned xl = r16 * (KDIM * 2) + ((lane & 3) ^ ((r16 >> 2) & 3)) * 16;
    const unsigned wl0 = r * (NS * 2) + ((lane & 7) ^ ((r >> 1) & 1)) * 16;
    const unsigned wl1 = (r + 8) * (NS * 2) + ((lane & 7) ^ (((r >> 1) & 1) | 4)) * 16;
    const int dsel = g % 7;
    struct Unit { u32x4 v[4]; char* dp; unsigned* fp; int idx; };
    auto fetch = [&](int i, Unit& U) {
      const int c = i / (PPC / 7);
      const int d = 1 + dsel, u = i % (PPC / 7);
      const int dest = (rank + d) & 7;
      char* dws = P.wsr[d];
      bool isw = false; int xu = u;
      if (d & 1) { if (u < 4) isw = true; else xu = 10 + (u - 4); }
      const int t = xu >> 1, h = xu & 1;
      const int dmt = t < 5 ? (rank >> 1) * 5 + t : 20 + (rank >> 1) * 3 + (t - 5);
      const char* sb = isw ? P.w + (size_t)(c * KC + u * 32) * (NS * 2) + (dest >> 1) * 128
                           : P.x + (size_t)(t * 32) * (KDIM * 2) + c * (KC * 2) + h * 128;
      const unsigned o0 = isw ? wl0 : xl, o1 = isw ? wl1 : xl + 16 * KDIM * 2;
      const unsigned o2 = isw ? wl0 + 16 * NS * 2 : xl + 64, o3 = isw ? wl1 + 16 * NS * 2 : xl + 64 + 16 * KDIM * 2;
      asm volatile("global_load_dwordx4 %0, %1, off" : "=v"(U.v[0]) : "v"(sb + o0) : "memory");
      asm volatile("global_load_dwordx4 %0, %1, off" : "=v"(U.v[1]) : "v"(sb + o1) : "memory");
      asm volatile("global_load_dwordx4 %0, %1, off" : "=v"(U.v[2]) : "v"(sb + o2) : "memory");
      asm volatile("global_load_dwordx4 %0, %1, off" : "=v"(U.v[3]) : "v"(sb + o3) : "memory");
      U.idx = i;
      U.dp = dws + (isw ? (unsigned)(XSTG_BYTES + (rank >> 1) * WT_BYTES) + c * WS_BYTES + u * 4096
                        : (unsigned)(dmt * XT_BYTES) + c * XS_BYTES + h * 4096) + lane * 16;
      U.fp = (unsigned*)(dws + FLAG_OFF) + (isw ? FW_W + ((rank >> 1) * NC + c) * 4 + u : FX_W + (dmt * NC + c) * 2 + h);
    };
    Unit UA, UB;
    const int g0 = g / 7, NPL = NP / 7 + (g % 7 < NP % 7 ? 1 : 0), NPAIRSL = NPAIRS / 7;
    unsigned* ticket = myf + TICKET_W + dsel * 64;
    fetch(g0, UA);
    fetch(g0 + NPL, UB);
    WAITV(4);
    asm volatile("" : "+v"(call));
    call = __builtin_amdgcn_readfirstlane(call) + 1;
    if (lane == 0) st4_sys(myf + CNTI(NWG + g), call);
    const size_t poff = (size_t)(call & 1) * PAR_BYTES;
    const unsigned tbase = (call - 1) * (unsigned)(NPAIRSL - NPL) - 2u * NPL;
    STAMP(6);
    int nxt = g0 + NPL;
    auto step = [&](Unit& U) -> bool {
      char* dp = U.dp + poff;
#pragma unroll
      for (int ks = 0; ks < (HALFU ? 2 : 4); ++ks) {
        st16_sys(dp + ks * 1024, U.v[ks]);
        if ((ks == 1 || ks == 2) && (TAILX == 0 || U.idx < NPAIRSL - TAILX)) WAITV(1);
      }
      unsigned tk = 0;
      const bool more = nxt < NPAIRSL;
      if (more && lane == 0) tk = __hip_atomic_fetch_add(ticket, 1u, __ATOMIC_RELAXED, __HIP_MEMORY_SCOPE_SYSTEM);
      WAITV(0);
      if (lane == 0) st4_sys(U.fp, call);
      if (!more) return false;
      nxt = (int)(__builtin_amdgcn_readfirstlane(tk) - tbase);
      if (nxt < NPAIRSL) fetch(nxt, U);
      return true;
    };
    while (true) {
      if (!step(UA)) break;
      if (!step(UB)) break;
    }
    STAMP(1);
    asm volatile("agk_code_end1:");
    return;
  }

  if (wv == 4) {
    unsigned call = 0;
    if (lane < 8) lds_st(ctl + lane, 0u);
    asm volatile("s_waitcnt lgkmcnt(0)" ::: "memory");
    __builtin_amdgcn_s_barrier();
    unsigned warmv = 0;
    {
      Tile T;
      get_tile(rank, b, T);
      if (lane == 1) warmv = ld4_sys(P.out[T.owner] + (size_t)(T.xsrc * MS + T.xt * 32) * (NS * 2) + T.ct * 128);
      if (lane == 2) warmv = ld4_sys((unsigned*)(P.ws[T.owner] + FLAG_OFF) + FY_W);
    }
    if (lane == 0) {
      call = ld4_sys(myf + CNTI(b)) + 1;
      st4_sys(myf + CNTI(b), call);
    }
    call = __builtin_amdgcn_readfirstlane(call);
    __builtin_amdgcn_fence(__ATOMIC_ACQUIRE, "");
    asm volatile("s_waitcnt vmcnt(0)" ::: "memory");
    if (lane == 0) lds_st(ctl + 0, call);
    asm volatile("" :: "v"(warmv));
    const unsigned long long t0 = __builtin_amdgcn_s_memrealtime();
    bool bad = false;
    {
      Tile T;
      get_tile(rank, b, T);
      const bool xrem = T.xsrc != rank, wrem = T.owner != rank;
      int rdy = 0;
      if (!xrem && !wrem) { rdy = NC; if (lane == 0) lds_st(ctl + 1, (unsigned)NC); }
      unsigned it = 0;
      while (rdy < NC) {
        unsigned f = call;
        const int cx = rdy + (lane >> 1), cw = rdy + ((lane - 8) >> 2);
        if (lane < 8) { if (xrem && cx < NC) f = ld4_sys(myf + FX_W + T.mt * 32 + rdy * 2 + lane); }
        else if (lane < 24) { if (wrem && cw < NC) f = ld4_sys(myf + FW_W + T.wslot * 64 + rdy * 4 + (lane - 8)); }
        unsigned tm = 0;
        if ((++it & 15) == 0) tm = __hip_atomic_load(tmo, __ATOMIC_RELAXED, __HIP_MEMORY_SCOPE_SYSTEM);
        const unsigned bl = (unsigned)__ballot(f >= call);
        unsigned bx = bl & 0xffu, bw = (bl >> 8) & 0xffffu;
        bx &= bx >> 1; bx &= 0x55u;
        bw &= bw >> 1; bw &= bw >> 2; bw &= 0x1111u;
        const int nrx = __builtin_ctz((~bx & 0x55u) | 0x100u) >> 1;
        const int nrw = __builtin_ctz((~bw & 0x1111u) | 0x10000u) >> 2;
        int nr = rdy + (nrx < nrw ? nrx : nrw);
        nr = nr < NC ? nr : NC;
        if (tm == call || ((it & 15) == 0 && __builtin_amdgcn_s_memrealtime() - t0 > DEADLINE)) {
          if (lane == 0) { __hip_atomic_store(tmo, call, __ATOMIC_RELAXED, __HIP_MEMORY_SCOPE_SYSTEM); lds_st(ctl + 4, 1u); }
          bad = true; nr = NC;
        }
        if (nr > rdy) {
          if (lane == 0) lds_st(ctl + 1, (unsigned)nr);
          rdy = nr;
        } else {
          __builtin_amdgcn_s_sleep(1);
        }
      }
    }
    STAMP(2);
    if (b < 8) {
      bool done = false;
      while (!done) {
        const unsigned fy = ld4_sys(myf + FY_W + b * 64 + lane);
        const unsigned tm = __hip_atomic_load(tmo, __ATOMIC_RELAXED, __HIP_MEMORY_SCOPE_SYSTEM);
        done = __all(fy >= call);
        if (!done) {
          if (tm == call || __builtin_amdgcn_s_memrealtime() - t0 > DEADLINE) {
            if (lane == 0) __hip_atomic_store(tmo, call, __ATOMIC_RELAXED, __HIP_MEMORY_SCOPE_SYSTEM);
            bad = true; done = true;
          } else __builtin_amdgcn_s_sleep(1);
        }
      }
      if (bad && lane == 0) {
        ((unsigned short*)P.out[rank])[(size_t)(rank * MS) * NS + b] = 0x7fc0;
      }
      STAMP(3);
    }
    asm volatile("agk_code_end2:");
    return;
  }

  const int wm = wv >> 1, wn = wv & 1;
  const unsigned lbase = (unsigned)(size_t)(__attribute__((address_space(3))) char*)smem;
  const int r8 = lane >> 3;
  __builtin_amdgcn_s_barrier();
  asm volatile("" ::: "memory");
  unsigned warm2 = 0;
  if ((int)(tid * 64u) < code_len) warm2 = *(const unsigned*)(code_pc + tid * 64u);
  unsigned call = lds_ld(ctl + 0);
  while (call == 0) { __builtin_amdgcn_s_sleep(0); call = lds_ld(ctl + 0); }
  asm volatile("" ::: "memory");
  asm volatile("" :: "v"(warm2));
  const char* sbase = myws + (size_t)(call & 1) * PAR_BYTES;

  const int m16 = lane & 15, g4 = lane >> 4, q4 = (lane & 15) >> 2, p4 = lane & 3;
  const unsigned xa0 = wv * 2048 + m16 * 64 + ((g4 ^ ((m16 >> 2) & 3)) * 16);
  const unsigned fwl = ((q4 >> 1) & 1) | ((g4 & 1) << 2);
  const unsigned wa_0 = wv * 4096 + g4 * 1024 + q4 * 128 + ((p4 ^ fwl) * 16) + ((p4 & 1) * 8);

  {
    Tile T;
    get_tile(rank, b, T);
    const bool xrem = T.xsrc != rank, wrem = T.owner != rank;
    const char* xg; unsigned xstride_c, xstride_ks;
    if (xrem) {
      xg = sbase + (size_t)T.mt * XT_BYTES + wv * 2048 + lane * 16; xstride_c = XS_BYTES; xstride_ks = 1024;
    } else {
      const int r16 = lane >> 2;
      const int c4 = (lane & 3) ^ ((r16 >> 2) & 3);
      xg = P.x + (size_t)(T.xt * 32 + r16) * (KDIM * 2) + wv * 64 + c4 * 16; xstride_c = KC * 2; xstride_ks = 16 * KDIM * 2;
    }
    const char* wg0; const char* wg1; unsigned wstride_c, wstride_k2;
    if (wrem) {
      wg0 = sbase + XSTG_BYTES + (size_t)T.wslot * WT_BYTES + wv * 4096 + lane * 16; wg1 = wg0 + 1024;
      wstride_c = WS_BYTES; wstride_k2 = 2048;
    } else {
      const int f0 = ((r8 >> 1) & 1), f1 = f0 | 4;
      wg0 = P.w + (size_t)(wv * 32 + r8) * (NS * 2) + T.ct * 128 + (((lane & 7) ^ f0) * 16);
      wg1 = P.w + (size_t)(wv * 32 + 8 + r8) * (NS * 2) + T.ct * 128 + (((lane & 7) ^ f1) * 16);
      wstride_c = KC * NS * 2; wstride_k2 = 16 * NS * 2;
    }
    f32x4 acc[2][2][2];
#pragma unroll
    for (int i = 0; i < 8; ++i) acc[i >> 2][(i >> 1) & 1][i & 1] = f32x4{0.f, 0.f, 0.f, 0.f};
    constexpr int cbase = 0;
    int issued = 0;
    for (int c = 0; c < NC; ++c) {
      int rdy = (int)lds_ld(ctl + 1) - cbase;
      while (issued <= c && rdy <= issued) {
        __builtin_amdgcn_s_sleep(CSLEEP); rdy = (int)lds_ld(ctl + 1) - cbase;
      }
      asm volatile("" ::: "memory");
      int lim = c + PFD;
      lim = lim < rdy ? lim : rdy;
      lim = lim < NC ? lim : NC;
      while (issued < lim) {
        const int sl = (cbase + issued) % NSLOT;
        const unsigned lw = lbase + LDS_W + sl * WS_BYTES + wv * 4096;
        const unsigned lx = lbase + LDS_X + sl * XS_BYTES + wv * 2048;
        const char* w0 = wg0 + (size_t)issued * wstride_c;
        const char* w1 = wg1 + (size_t)issued * wstride_c;
        const char* x0 = xg + (size_t)issued * xstride_c;
        {
        dma16<LD_AUX>(w0, lw);
        dma16<LD_AUX>(w1, lw + 1024);
        dma16<LD_AUX>(w0 + wstride_k2, lw + 2048);
        dma16<LD_AUX>(w1 + wstride_k2, lw + 3072);
        dma16<LD_AUX>(x0, lx);
        dma16<LD_AUX>(x0 + xstride_ks, lx + 1024);
        }
        ++issued;
      }
      switch (issued - c - 1) {
        case 0: WAITV(0); break;
        case 1: WAITV(6); break;
        case 2: WAITV(12); break;
        case 3: WAITV(18); break;
        default: WAITV(24); break;
      }
      {
        const int sl = c % NSLOT;
        const unsigned ax = lbase + LDS_X + sl * XS_BYTES + xa0;
        const unsigned aw = lbase + LDS_W + sl * WS_BYTES + wa_0;
        const unsigned aw00 = aw, aw01 = aw ^ 8, aw10 = aw ^ 64, aw11 = aw ^ 72;
        u32x4 xf[2]; u32x2 wf[2][2][2];
        asm volatile(
          "ds_read_b128 %0, %10\n\t"
          "ds_read_b64_tr_b16 %2, %11\n\t"
          "ds_read_b64_tr_b16 %3, %11 offset:512\n\t"
          "ds_read_b64_tr_b16 %4, %12\n\t"
          "ds_read_b64_tr_b16 %5, %12 offset:512\n\t"
          "ds_read_b128 %1, %10 offset:1024\n\t"
          "ds_read_b64_tr_b16 %6, %13\n\t"
          "ds_read_b64_tr_b16 %7, %13 offset:512\n\t"
          "ds_read_b64_tr_b16 %8, %14\n\t"
          "ds_read_b64_tr_b16 %9, %14 offset:512\n\t"
          "s_waitcnt lgkmcnt(0)"
          : "=&v"(xf[0]), "=&v"(xf[1]), "=&v"(wf[0][0][0]), "=&v"(wf[0][0][1]), "=&v"(wf[0][1][0]), "=&v"(wf[0][1][1]),
            "=&v"(wf[1][0][0]), "=&v"(wf[1][0][1]), "=&v"(wf[1][1][0]), "=&v"(wf[1][1][1])
          : "v"(ax), "v"(aw00), "v"(aw01), "v"(aw10), "v"(aw11) : "memory");
#pragma unroll
        for (int i = 0; i < 4; ++i) {
          const int n_ = i >> 1, t_ = i & 1;
          const u32x4 a = {wf[n_][t_][0][0], wf[n_][t_][0][1], wf[n_][t_][1][0], wf[n_][t_][1][1]};
          acc[0][n_][t_] = __builtin_amdgcn_mfma_f32_16x16x32_bf16(__builtin_bit_cast(bf16x8, a), __builtin_bit_cast(bf16x8, xf[0]), acc[0][n_][t_], 0, 0, 0);
          acc[1][n_][t_] = __builtin_amdgcn_mfma_f32_16x16x32_bf16(__builtin_bit_cast(bf16x8, a), __builtin_bit_cast(bf16x8, xf[1]), acc[1][n_][t_], 0, 0, 0);
        }
      }
    }
    f32x4 acc0, acc1;
    {
      f32x4* mine = (f32x4*)(smem + LDS_W + wv * 4096) + lane;
#pragma unroll
      for (int i = 0; i < 8; ++i) mine[(i >> 2) * (WS_BYTES / 16) + (i & 3) * 64] = acc[i >> 2][(i >> 1) & 1][i & 1];
      asm volatile("s_waitcnt lgkmcnt(0)" ::: "memory");
      if (lane == 0) __hip_atomic_fetch_add(ctl + 3, 1u, __ATOMIC_RELAXED, __HIP_MEMORY_SCOPE_WORKGROUP);
      while (lds_ld(ctl + 3) < 4u) { __builtin_amdgcn_s_sleep(0); }
      asm volatile("" ::: "memory");
      const int cg = lane & 7, srcl = (wv & 1) * 8 + (lane >> 3) + 16 * (cg & 3);
      const f32x4* src = (const f32x4*)(smem + LDS_W + wm * WS_BYTES + (cg >> 2) * 2048) + srcl;
      acc0 = src[0]; acc1 = src[64];
#pragma unroll
      for (int q = 1; q < 4; ++q) { acc0 += src[q * 256]; acc1 += src[q * 256 + 64]; }
    }
    const bool bad = lds_ld(ctl + 4) != 0;
    const f32x4 lo = (lane & 1) ? acc1 : acc0, hi = (lane & 1) ? acc0 : acc1;
    u32x4 o;
    o[0] = pack_relu(lo[0], lo[1]); o[1] = pack_relu(lo[2], lo[3]);
    o[2] = pack_relu(hi[0], hi[1]); o[3] = pack_relu(hi[2], hi[3]);
    if (bad) { o[0] = o[1] = o[2] = o[3] = 0x7fc07fc0u; }
    const int grow = T.xsrc * MS + T.xt * 32 + wv * 8 + (lane >> 3);
    char* op = P.out[T.owner] + (size_t)grow * (NS * 2) + T.ct * 128 + (lane & 7) * 16;
    if (!wrem) {
      st16_sys(op, o);
    } else {
      st16_sys(op, o);
      WAITV(0);
      if (lane == 0) st4_sys((unsigned*)(P.ws[T.owner] + FLAG_OFF) + FY_W + ((rank >> 1) * 32 + T.mt) * 4 + wv, call);
    }
    if (wv == 0) STAMP(4);
    asm volatile("agk_code_end0:");
  }
}

}

extern "C" void kernel_launch(int rank, int nranks, ncclComm_t comm,
                              hipStream_t stream,
                              void* const* d_in, const int* in_sizes, int n_in,
                              void* d_out, int out_size,
                              const int* in_dtypes, int out_dtype,
                              void* const* d_out_all,
                              void* const* d_ws_all, size_t ws_bytes) {
  if (nranks != NR || ws_bytes < WS_NEED || n_in < 2 || (unsigned)rank >= (unsigned)NR) {
    fprintf(stderr, "unsupported configuration\n");
    return;
  }
  Params p;
  p.x = (const char*)d_in[0];
  p.w = (const char*)d_in[1];
  for (int i = 0; i < NR; ++i) { p.out[i] = (char*)d_out_all[i]; p.ws[i] = (char*)d_ws_all[i]; }
  for (int i = 0; i < NR; ++i) p.wsr[i] = (char*)d_ws_all[(rank + i) & 7];
  p.myws = (char*)d_ws_all[rank];
  p.rank = rank;
  ag_gemm_kernel<<<NWG, NTHREADS, 0, stream>>>(p);
  HIP_CHECK(hipGetLastError());
}
```
